```python
import numpy as np

kernel_name = "gpu_gemm_ar_m2048_k2048_n2048_bf16_relu"


ABSMAX_RTOL = 0.008

COMPUTE_DTYPE = "bfloat16"
EPILOGUE = "relu"
IN_QUANT = None
OUT_QUANT = None
IN_DTYPES = {"x": "bfloat16", "w_mat": "bfloat16"}
WORLD_SIZE = 8

def _to_bf16(x):
    u = np.ascontiguousarray(x, dtype=np.float32).view(np.uint32)
    return ((u + 0x7FFF + ((u >> 16) & 1)) & 0xFFFF0000).view(np.float32)

def setup_inputs():

    x = _to_bf16(np.random.randn(2048, 2048).astype(np.float32))
    w_mat = _to_bf16((np.random.randn(2048, 2048) / np.sqrt(2048)).astype(np.float32))
    return {"x": x, "w_mat": w_mat}

def _epilogue(y):
    return np.maximum(y, 0.0).astype(np.float32)

def reference(x, w_mat):
    y = (x @ w_mat).astype(np.float32)
    return _epilogue(y)
```

```cpp
#include <hip/hip_runtime.h>
#include <rccl/rccl.h>
#include <cstdio>
#include <cstdint>

#define HIP_CHECK(expr) do { hipError_t _e = (expr); if (_e != hipSuccess) { \
    fprintf(stderr, "HIP error %s at %s:%d\n", hipGetErrorString(_e), __FILE__, __LINE__); \
    return; } } while (0)

namespace {

typedef unsigned u32;
typedef unsigned long long u64;
typedef u32 u32x4 __attribute__((ext_vector_type(4)));
typedef u32 u32x2 __attribute__((ext_vector_type(2)));
typedef float f32x16 __attribute__((ext_vector_type(16)));
typedef float f32x4 __attribute__((ext_vector_type(4)));
typedef float f32x2 __attribute__((ext_vector_type(2)));
typedef __bf16 bf16x8 __attribute__((ext_vector_type(8)));
typedef __bf16 bf16x2 __attribute__((ext_vector_type(2)));
typedef __attribute__((address_space(1))) u64 gu64;
typedef __attribute__((address_space(1))) u32 gu32;

constexpr int NR = 8;
constexpr int MM = 2048, NN = 2048, KL = 256;
constexpr int NWG = 256, NT = 512;
constexpr int PIN = 4;

constexpr u32 OFF_CNT = 0;
constexpr u32 OFF_TMO = OFF_CNT + NWG * 128;
constexpr u32 OFF_WF = 65536;
constexpr u32 OFF_XF = OFF_WF + 64 * 128;
constexpr u32 OFF_OF = OFF_XF + 256 * 128;
constexpr u32 OFF_DBG = OFF_OF + 2048 * 128;
constexpr u32 OFF_WD = 1u << 20;
constexpr u32 OFF_XD = OFF_WD + 8 * 8 * 8 * PIN * 1024;
constexpr u32 WS_NEED = OFF_XD + 8 * 32 * 4 * PIN * 1024;
static_assert(OFF_DBG + 256 * 8 * 16 * 8 <= OFF_WD, "layout");

constexpr u64 DEADLINE = 20000000ull;
constexpr int AUX_SYS = 17;

constexpr int LDS_RED = 0;
constexpr int LDS_TILE = 65536;
constexpr int LDS_BAD = LDS_TILE + 4096;
constexpr int LDS_BYTES = LDS_BAD + 64;

struct Params {
  const uint16_t* x;
  const uint16_t* w;
  char* ws[NR];
  char* out[NR];
  int rank;
  int pad;
};

__device__ __forceinline__ __amdgpu_buffer_rsrc_t mk_rsrc(const void* p, u32 bytes) {
  return __builtin_amdgcn_make_buffer_rsrc((void*)p, 0, bytes, 0x00020000);
}

#define STAMP(i) do {} while (0)

__device__ __forceinline__ bool wait_flags(char* myws, u32 off, int n, u32 call, int lane, u64& val) {
  gu64* addr = (gu64*)(myws + (lane < n ? off + lane * 8 : OFF_TMO));
  const u64 t0 = __builtin_amdgcn_s_memrealtime();
  bool ok = true;
  for (;;) {
    u64 v = (u64)call;
    if (lane <= n) v = __hip_atomic_load(addr, __ATOMIC_RELAXED, __HIP_MEMORY_SCOPE_SYSTEM);
    val = v;
    const u64 pend = __builtin_amdgcn_ballot_w64(lane < n && (u32)v < call);
    if (pend == 0) break;
    const u64 tm = __builtin_amdgcn_ballot_w64(lane == n && (u32)v == call);
    if (tm != 0 || __builtin_amdgcn_s_memrealtime() - t0 > DEADLINE) {
      if (lane == 0) __hip_atomic_store((gu64*)(myws + OFF_TMO), (u64)call, __ATOMIC_RELAXED, __HIP_MEMORY_SCOPE_AGENT);
      ok = false;
      break;
    }
    __builtin_amdgcn_s_sleep(2);
  }
  asm volatile("" ::: "memory");
  return ok;
}

__device__ __forceinline__ u32 cvt_pk(float a, float b_) {
  f32x2 f = {a, b_};
  return __builtin_bit_cast(u32, __builtin_convertvector(f, bf16x2));
}

__global__ __launch_bounds__(NT, 2) void gemm_ar_kernel(Params P) {
  extern __shared__ __attribute__((aligned(16))) char smem[];
  const int tid = threadIdx.x;
  const int lane = tid & 63;
  const int wave = __builtin_amdgcn_readfirstlane(tid >> 6);
  const int b = blockIdx.x;
  const int rank = P.rank;
  const int xcd = b & 7, idx = b >> 3, cls = idx >> 3, g = cls * 8 + xcd, c = idx & 7;
  char* myws = P.ws[rank];
  const int h = lane >> 5, l31 = lane & 31;
  STAMP(0);

  const u32 call = __hip_atomic_load((gu32*)(myws + OFF_CNT + b * 128), __ATOMIC_RELAXED, __HIP_MEMORY_SCOPE_AGENT) + 1;
  bool ok = true;

  if (wave == ((rank + 1) & 7)) {
    const int cgl = b >> 3, u8 = b & 7;
    const u32* src = (const u32*)(P.w + (size_t)(128 * h + 16 * u8) * NN + 64 * cgl + 2 * l31);
    u32 r[16];
#pragma unroll
    for (int k = 0; k < 16; ++k) r[k] = src[k * (NN / 2)];
    u32x4 pc[4];
#pragma unroll
    for (int e = 0; e < 2; ++e)
#pragma unroll
      for (int q = 0; q < 4; ++q) {
        pc[e][q] = __builtin_amdgcn_perm(r[8 * e + 2 * q + 1], r[8 * e + 2 * q], 0x05040100u);
        pc[2 + e][q] = __builtin_amdgcn_perm(r[8 * e + 2 * q + 1], r[8 * e + 2 * q], 0x07060302u);
      }
    const int jd = cgl >> 3, cd = cgl & 7;
    const u32 doff = OFF_WD + (u32)(((rank * 8 + cd) * 8 + u8) * PIN) * 1024u + lane * 16;
#pragma unroll
    for (int d = 0; d < 2; ++d) {
      const int dst = jd + 4 * d;
      __amdgpu_buffer_rsrc_t rs = mk_rsrc(P.ws[dst], WS_NEED);
#pragma unroll
      for (int p = 0; p < 4; ++p) __builtin_amdgcn_raw_buffer_store_b128(pc[p], rs, doff + p * 1024, 0, AUX_SYS);
    }
    asm volatile("s_waitcnt vmcnt(0)" ::: "memory");
    if (lane < 2) {
      char* dws = P.ws[jd + 4 * lane];
      __hip_atomic_store((gu64*)(dws + OFF_WF + (rank * 8 + cd) * 128 + u8 * 8), (u64)call, __ATOMIC_RELAXED, __HIP_MEMORY_SCOPE_SYSTEM);
    }
  }
  if (wave == rank) {
    const int ih = c >> 2, u = c & 3;
    const u32x4* src = (const u32x4*)(P.x + (size_t)(1024 * ih + 32 * g + l31) * KL + 128 * h + 32 * u);
    u32x4 pc[4];
#pragma unroll
    for (int p = 0; p < 4; ++p) pc[p] = src[p];
    const u32 doff = OFF_XD + (u32)(((rank * 32 + g) * 4 + u) * PIN) * 1024u + lane * 16;
#pragma unroll
    for (int d = 0; d < 4; ++d) {
      const int dst = 4 * ih + d;
      __amdgpu_buffer_rsrc_t rs = mk_rsrc(P.ws[dst], WS_NEED);
#pragma unroll
      for (int p = 0; p < 4; ++p) __builtin_amdgcn_raw_buffer_store_b128(pc[p], rs, doff + p * 1024, 0, AUX_SYS);
    }
    asm volatile("s_waitcnt vmcnt(0)" ::: "memory");
    if (lane < 4) {
      char* dws = P.ws[4 * ih + lane];
      __hip_atomic_store((gu64*)(dws + OFF_XF + (rank * 32 + g) * 128 + u * 8), (u64)call, __ATOMIC_RELAXED, __HIP_MEMORY_SCOPE_SYSTEM);
    }
  }
  STAMP(1);

  __amdgpu_buffer_rsrc_t rsme = mk_rsrc(myws, WS_NEED);
  u32x4 wf[2][16];
  {
    u64 fv;
    ok = wait_flags(myws, OFF_WF + (wave * 8 + c) * 128, 8, call, lane, fv) && ok;
    STAMP(2);
    const u32 off = OFF_WD + (u32)((wave * 8 + c) * 8 * PIN) * 1024u + lane * 16;
#pragma unroll
    for (int u8 = 0; u8 < 8; ++u8)
#pragma unroll
      for (int p = 0; p < 4; ++p)
        wf[p >> 1][2 * u8 + (p & 1)] = __builtin_amdgcn_raw_buffer_load_b128(rsme, off + (u8 * PIN + p) * 1024, 0, AUX_SYS);
  }
  STAMP(3);
  f32x16 acc0 = {}, acc1 = {};
  {
    u64 fv;
    ok = wait_flags(myws, OFF_XF + (wave * 32 + g) * 128, 4, call, lane, fv) && ok;
    STAMP(4);
    const u32 off = OFF_XD + (u32)((wave * 32 + g) * 4 * PIN) * 1024u + lane * 16;
    u32x4 xf[16];
#pragma unroll
    for (int m = 0; m < 16; ++m) xf[m] = __builtin_amdgcn_raw_buffer_load_b128(rsme, off + m * 1024, 0, AUX_SYS);
    __builtin_amdgcn_sched_barrier(0);
#pragma unroll
    for (int m = 0; m < 16; ++m) {
      acc0 = __builtin_amdgcn_mfma_f32_32x32x16_bf16(__builtin_bit_cast(bf16x8, xf[m]), __builtin_bit_cast(bf16x8, wf[0][m]), acc0, 0, 0, 0);
      acc1 = __builtin_amdgcn_mfma_f32_32x32x16_bf16(__builtin_bit_cast(bf16x8, xf[m]), __builtin_bit_cast(bf16x8, wf[1][m]), acc1, 0, 0, 0);
    }
  }
  STAMP(5);
  {
    char* red = smem + LDS_RED + wave * 8192;
#pragma unroll
    for (int r = 0; r < 16; ++r) {
      const int row = (r & 3) + 8 * (r >> 2) + 4 * h;
      const int off = (row * 256 + l31 * 8) ^ (((row >> 3) & 1) << 7);
      *(f32x2*)(red + off) = f32x2{acc0[r], acc1[r]};
    }
    u32 tm = 0;
    if (lane == 0) tm = (u32)__hip_atomic_load((gu64*)(myws + OFF_TMO), __ATOMIC_RELAXED, __HIP_MEMORY_SCOPE_AGENT);
    const bool bad = !ok || __builtin_amdgcn_readfirstlane(tm) == call;
    if (lane == 0) ((u32*)(smem + LDS_BAD))[wave] = bad ? 1u : 0u;
  }
  __syncthreads();
  {
    const int cg = lane >> 3, q = (lane >> 1) & 3, hf = lane & 1;
    const int row = 8 * q + wave;
    const int off = (row * 256 + cg * 32 + hf * 16) ^ ((q & 1) << 7);
    f32x4 s = *(const f32x4*)(smem + LDS_RED + off);
#pragma unroll
    for (int w2 = 1; w2 < 8; ++w2) s += *(const f32x4*)(smem + LDS_RED + w2 * 8192 + off);
    const u32x4 bw0 = *(const u32x4*)(smem + LDS_BAD), bw1 = *(const u32x4*)(smem + LDS_BAD + 16);
    const bool bad = (bw0[0] | bw0[1] | bw0[2] | bw0[3] | bw1[0] | bw1[1] | bw1[2] | bw1[3]) != 0;
    float v[4];
#pragma unroll
    for (int e = 0; e < 4; ++e) v[e] = !(s[e] <= 0.f) ? s[e] : 0.f;
    u32x2 o = {cvt_pk(v[0], v[1]), cvt_pk(v[2], v[3])};
    if (bad) o = u32x2{0x7fc07fc0u, 0x7fc07fc0u};
    *(u32x2*)(smem + LDS_TILE + row * 128 + cg * 16 + hf * 8) = o;
  }
  __syncthreads();
  STAMP(6);
  {
    const int i = rank >> 2, j = rank & 3;
    __amdgpu_buffer_rsrc_t rso = mk_rsrc(P.out[wave], MM * NN * 2);
#pragma unroll
    for (int q = 0; q < 4; ++q) {
      const int row = 8 * q + (lane >> 3);
      const u32x4 v = *(const u32x4*)(smem + LDS_TILE + row * 128 + (lane & 7) * 16);
      const u32 off = (u32)(1024 * i + 32 * g + row) * (NN * 2) + (512 * j + 64 * c) * 2 + (lane & 7) * 16;
      __builtin_amdgcn_raw_buffer_store_b128(v, rso, off, 0, AUX_SYS);
    }
    asm volatile("s_waitcnt vmcnt(0)" ::: "memory");
    if (lane == 0) {
      char* dws = P.ws[wave];
      __hip_atomic_store((gu64*)(dws + OFF_OF + (rank * 256 + g * 8 + c) * 128), (u64)call, __ATOMIC_RELAXED, __HIP_MEMORY_SCOPE_SYSTEM);
    }
  }
  STAMP(7);
  {
    u64 fv;
    const bool ok2 = wait_flags(myws, OFF_OF + (wave * 256 + g * 8 + c) * 128, 1, call, lane, fv);
    if (!ok2 && lane == 0) {
      const int i = rank >> 2, j = rank & 3;
      ((uint16_t*)P.out[rank])[(size_t)(1024 * i + 32 * g) * NN + 512 * j + 64 * c] = 0x7fc0;
    }
  }
  if (tid == 0) __hip_atomic_store((gu32*)(myws + OFF_CNT + b * 128), call, __ATOMIC_RELAXED, __HIP_MEMORY_SCOPE_AGENT);
  STAMP(8);
}

}

extern "C" void kernel_launch(int rank, int nranks, ncclComm_t comm, hipStream_t stream,
                              void* const* d_in, const int* in_sizes, int n_in,
                              void* d_out, int out_size, const int* in_dtypes, int out_dtype,
                              void* const* d_out_all, void* const* d_ws_all, size_t ws_bytes) {
  if (nranks != NR || ws_bytes < WS_NEED || (unsigned)rank >= (unsigned)NR) {
    fprintf(stderr, "unexpected configuration\n");
    return;
  }
  static bool init[NR] = {};
  if (!init[rank]) {
    HIP_CHECK(hipFuncSetAttribute((const void*)gemm_ar_kernel, hipFuncAttributeMaxDynamicSharedMemorySize, LDS_BYTES));
    init[rank] = true;
  }
  Params p;
  p.x = (const uint16_t*)d_in[0];
  p.w = (const uint16_t*)d_in[1];
  for (int r = 0; r < NR; ++r) {
    p.ws[r] = (char*)d_ws_all[r];
    p.out[r] = (char*)d_out_all[r];
  }
  p.rank = rank;
  p.pad = 0;
  hipLaunchKernelGGL(gemm_ar_kernel, dim3(NWG), dim3(NT), LDS_BYTES, stream, p);
}
```

```cpp
#include <hip/hip_runtime.h>
#include <rccl/rccl.h>
#include <cstdio>
#include <cstdint>
#include <cstdlib>

#define HIP_CHECK(expr) do { hipError_t _e = (expr); if (_e != hipSuccess) { \
    fprintf(stderr, "HIP error %s at %s:%d\n", hipGetErrorString(_e), __FILE__, __LINE__); \
    return; } } while (0)

namespace {

typedef unsigned u32;
typedef unsigned long long u64;
typedef u32 u32x4 __attribute__((ext_vector_type(4)));
typedef u32 u32x2 __attribute__((ext_vector_type(2)));
typedef float f32x16 __attribute__((ext_vector_type(16)));
typedef float f32x4 __attribute__((ext_vector_type(4)));
typedef float f32x2 __attribute__((ext_vector_type(2)));
typedef __bf16 bf16x8 __attribute__((ext_vector_type(8)));
typedef __bf16 bf16x2 __attribute__((ext_vector_type(2)));
typedef unsigned short u16x2 __attribute__((ext_vector_type(2)));
typedef __attribute__((address_space(1))) u64 gu64;
typedef __attribute__((address_space(1))) u32 gu32;

constexpr int NR = 8;
constexpr int MM = 2048, NN = 2048, KL = 256;
constexpr int NWG = 256, NT = 512;

constexpr u32 OFF_CNT = 0;
constexpr u32 OFF_TMO = OFF_CNT + NWG * 128;
constexpr u32 OFF_WF = 65536;
constexpr u32 OFF_XF = OFF_WF + 64 * 128;
constexpr u32 OFF_OF = OFF_XF + 256 * 128;
constexpr u32 OFF_DBG = OFF_OF + 2048 * 128;
constexpr u32 UNIT_B = 4096;
constexpr u32 OFF_WD = 1u << 20;
constexpr u32 OFF_XD = OFF_WD + 8 * 8 * 8 * UNIT_B;
constexpr u32 OT_B = 4608;
constexpr u32 OT_OVF = 3328, OT_BM = 4352;
constexpr u32 OFF_OD = OFF_XD + 8 * 32 * 4 * UNIT_B;
constexpr u32 WS_NEED = OFF_OD + 2048 * OT_B;
static_assert(OFF_DBG + 256 * 8 * 16 * 8 <= OFF_WD, "layout");

constexpr u64 DEADLINE = 20000000ull;
constexpr int AUX_SYS = 17;

#define TW0 130
#define DW 50
#define TX0 530
#define DX 27
#define TO0 1350
#define DO 30
#define LW 150
#define LX 150
#define LO 150
#define PSLEEP 4
#define OUT_AUX 16
#define GWFIRST 14

constexpr int LDS_RED = 0;
constexpr int UAREA = 4352;
constexpr int LDS_STR = 65536;
constexpr int LDS_MSK = LDS_STR + 4352;
constexpr int LDS_INFO = LDS_MSK + 256;
constexpr int LDS_BAD = LDS_INFO + 32;
constexpr int LDS_LUT = LDS_BAD + 32;
constexpr int LDS_BYTES = LDS_LUT + 2048;
static_assert(8 * UAREA <= 65536, "uarea");

struct Params {
  const uint16_t* x;
  const uint16_t* w;
  char* ws[NR];
  char* out;
  int rank;
  int pad;
};

__device__ __forceinline__ __amdgpu_buffer_rsrc_t mk_rsrc(const void* p, u32 bytes) {
  return __builtin_amdgcn_make_buffer_rsrc((void*)p, 0, bytes, 0x00020000);
}
__device__ __forceinline__ u32 perm(u32 hi, u32 lo, u32 sel) { return __builtin_amdgcn_perm(hi, lo, sel); }
__device__ __forceinline__ u32 rotr4(u32 v) { return __builtin_amdgcn_alignbit(v, v, 4); }
__device__ __forceinline__ u32 rotl4(u32 v) { return __builtin_amdgcn_alignbit(v, v, 28); }
__device__ __forceinline__ u32 pkmax(u32 a, u32 b) {
  return __builtin_bit_cast(u32, __builtin_elementwise_max(__builtin_bit_cast(u16x2, a), __builtin_bit_cast(u16x2, b)));
}
template <int CTRL, int RM>
__device__ __forceinline__ u32 dpp0(u32 v) { return (u32)__builtin_amdgcn_update_dpp(0, (int)v, CTRL, RM, 0xf, false); }
__device__ __forceinline__ u32 wave_max(u32 v) {
  v = max(v, dpp0<0x111, 0xf>(v));
  v = max(v, dpp0<0x112, 0xf>(v));
  v = max(v, dpp0<0x114, 0xf>(v));
  v = max(v, dpp0<0x118, 0xf>(v));
  v = max(v, dpp0<0x142, 0xa>(v));
  v = max(v, dpp0<0x143, 0xc>(v));
  return (u32)__builtin_amdgcn_readlane((int)v, 63);
}
__device__ __forceinline__ u32 wave_scan_incl(u32 v) {
  v += dpp0<0x111, 0xf>(v);
  v += dpp0<0x112, 0xf>(v);
  v += dpp0<0x114, 0xf>(v);
  v += dpp0<0x118, 0xf>(v);
  v += dpp0<0x142, 0xa>(v);
  v += dpp0<0x143, 0xc>(v);
  return v;
}

#define STAMP(i) do {} while (0)
#define STAMP2(i) do {} while (0)

__device__ __forceinline__ bool wait_flags(char* myws, u32 off, bool act, u32 call, int lane, u64& val, u64 tstart, u32 notbefore) {
  gu64* addr = (gu64*)(myws + off);
  while (__builtin_amdgcn_s_memrealtime() - tstart < (u64)notbefore) __builtin_amdgcn_s_sleep(2);
  const u64 t0 = __builtin_amdgcn_s_memrealtime();
  bool ok = true;
  for (;;) {
    u64 v = (u64)call;
    if (act) v = __hip_atomic_load(addr, __ATOMIC_RELAXED, __HIP_MEMORY_SCOPE_SYSTEM);
    val = v;
    const u64 pend = __builtin_amdgcn_ballot_w64(act && (u32)v < call);
    if (pend == 0) break;
    const u64 el = __builtin_amdgcn_s_memrealtime() - t0;
    if (el > 50000) {
      const u32 tm = (u32)__hip_atomic_load((gu64*)(myws + OFF_TMO), __ATOMIC_RELAXED, __HIP_MEMORY_SCOPE_AGENT);
      if (__builtin_amdgcn_readfirstlane(tm) == call || el > DEADLINE) {
        if (lane == 0) __hip_atomic_store((gu64*)(myws + OFF_TMO), (u64)call, __ATOMIC_RELAXED, __HIP_MEMORY_SCOPE_AGENT);
        ok = false;
        break;
      }
    }
    __builtin_amdgcn_s_sleep(PSLEEP);
  }
  asm volatile("" ::: "memory");
  return ok;
}
__device__ __forceinline__ u32 read_tmo(char* myws) {
  return (u32)__hip_atomic_load((gu64*)(myws + OFF_TMO), __ATOMIC_RELAXED, __HIP_MEMORY_SCOPE_AGENT);
}

__device__ __forceinline__ u32 cvt_pk(float a, float b_) {
  f32x2 f = {a, b_};
  return __builtin_bit_cast(u32, __builtin_convertvector(f, bf16x2));
}

struct Packed { u32x4 la, lb, n, ovf; u64 bad; };

__device__ __forceinline__ void pack_hi(const u32 (&ha)[4], const u32 (&hb)[4], u32 lo, bool lane_on, Packed& pk) {
  const u32 lo4 = lo * 0x01010101u;
  u32 acc = 0x80808080u;
#pragma unroll
  for (int j = 0; j < 4; ++j) {
    const u32 da = (ha[j] | 0x80808080u) - lo4;
    const u32 db = (hb[j] | 0x80808080u) - lo4;
    acc &= da & db;
    const u32 xa = (ha[j] & 0x80808080u) | (da & 0x7f7f7f7fu);
    const u32 xb = (hb[j] & 0x80808080u) | (db & 0x7f7f7f7fu);
    pk.n[j] = xa | rotr4(xb);
  }
  const bool bad = lane_on && acc != 0x80808080u;
  pk.bad = __builtin_amdgcn_ballot_w64(bad);
  if (pk.bad != 0) {
#pragma unroll
    for (int j = 0; j < 4; ++j) {
      pk.n[j] = bad ? ha[j] : pk.n[j];
      pk.ovf[j] = hb[j];
    }
  }
}
__device__ __forceinline__ void unpack_hi(const u32x4& n, u32 lo4, u32 (&ha)[4], u32 (&hb)[4]) {
#pragma unroll
  for (int j = 0; j < 4; ++j) {
    ha[j] = (n[j] & 0x87878787u) + lo4;
    hb[j] = (rotl4(n[j]) & 0x87878787u) + lo4;
  }
}
__device__ __forceinline__ u32 lo_from_max(u32 m16) {
  const u32 e7 = m16 >> 8;
  return e7 > 7 ? e7 - 7 : 0;
}

__device__ __forceinline__ void timer_wait(u64 tstart, u32 dly) {
  while (__builtin_amdgcn_s_memrealtime() - tstart < (u64)dly) __builtin_amdgcn_s_sleep(1);
}

template <int ND>
__device__ __forceinline__ void send_unit(const Params& P, const Packed& pk, u32 lo, int dst0, int dstep, u32 doff,
                                          u32 foff, u32 moff, u32 call, int lane, u64 tstart, u32 dly) {
  STAMP2(10 + (ND == 4 ? 2 : 0));
  timer_wait(tstart, dly);
  STAMP2(14 + (ND == 4 ? 1 : 0));
#pragma unroll
  for (int d = 0; d < ND; ++d) {
    __amdgpu_buffer_rsrc_t rs = mk_rsrc(P.ws[dst0 + d * dstep], WS_NEED);
    __builtin_amdgcn_raw_buffer_store_b128(pk.la, rs, doff + lane * 16, 0, AUX_SYS);
    __builtin_amdgcn_raw_buffer_store_b128(pk.lb, rs, doff + 1024 + lane * 16, 0, AUX_SYS);
    __builtin_amdgcn_raw_buffer_store_b128(pk.n, rs, doff + 2048 + lane * 16, 0, AUX_SYS);
  }
  if (pk.bad != 0) {
#pragma unroll
    for (int d = 0; d < ND; ++d) {
      __amdgpu_buffer_rsrc_t rs = mk_rsrc(P.ws[dst0 + d * dstep], WS_NEED);
      if ((pk.bad >> lane) & 1) __builtin_amdgcn_raw_buffer_store_b128(pk.ovf, rs, doff + 3072 + lane * 16, 0, AUX_SYS);
      if (lane == 0) __builtin_amdgcn_raw_buffer_store_b64(u32x2{(u32)pk.bad, (u32)(pk.bad >> 32)}, rs, moff, 0, AUX_SYS);
    }
  }
  asm volatile("s_waitcnt vmcnt(0)" ::: "memory");
  STAMP2(11 + (ND == 4 ? 2 : 0));
  const u32x2 f = {call, lo | (pk.bad != 0 ? 0x80u : 0u)};
  if (lane == 0) {
#pragma unroll
    for (int d = 0; d < ND; ++d)
      __builtin_amdgcn_raw_buffer_store_b64(f, mk_rsrc(P.ws[dst0 + d * dstep], WS_NEED), foff, 0, AUX_SYS);
  }
}

__device__ __forceinline__ void send_x(const Params& P, int g, int c, u32 call, int lane, u64 tstart, u32 dly) {
  const int rank = P.rank, h = lane >> 5, l31 = lane & 31;
  const int ih = c >> 2, u = c & 3;
  const u32x4* src = (const u32x4*)(P.x + (size_t)(1024 * ih + 32 * g + l31) * KL + 128 * h + 32 * u);
  u32 r[16];
#pragma unroll
  for (int p = 0; p < 4; ++p) {
    const u32x4 t = src[p];
#pragma unroll
    for (int q = 0; q < 4; ++q) r[4 * p + q] = t[q];
  }
  u32 m = 0;
#pragma unroll
  for (int t = 0; t < 16; ++t) m = pkmax(m, r[t] & 0x7fff7fffu);
  const u32 lo = lo_from_max(wave_max(max(m & 0xffffu, m >> 16)));
  Packed pk;
  u32 ha[4], hb[4];
#pragma unroll
  for (int j = 0; j < 4; ++j) {
    pk.la[j] = perm(r[2 * j + 1], r[2 * j], 0x06040200u);
    ha[j] = perm(r[2 * j + 1], r[2 * j], 0x07050301u);
    pk.lb[j] = perm(r[8 + 2 * j + 1], r[8 + 2 * j], 0x06040200u);
    hb[j] = perm(r[8 + 2 * j + 1], r[8 + 2 * j], 0x07050301u);
  }
  pack_hi(ha, hb, lo, true, pk);
  const u32 fl = OFF_XF + (rank * 32 + g) * 128;
  send_unit<4>(P, pk, lo, 4 * ih, 1, OFF_XD + (u32)((rank * 32 + g) * 4 + u) * UNIT_B, fl + u * 8, fl + 32 + u * 8, call, lane, tstart, dly);
}

__device__ __forceinline__ void send_w(const Params& P, int b, u32 call, int lane, u64 tstart, u32 dly) {
  const int rank = P.rank, h = lane >> 5, l31 = lane & 31;
  const int cgl = b >> 3, u8 = b & 7;
  const u32* src = (const u32*)(P.w + (size_t)(128 * h + 16 * u8) * NN + 64 * cgl + 2 * l31);
  u32 r[16];
#pragma unroll
  for (int k = 0; k < 16; ++k) r[k] = src[k * (NN / 2)];
  u32 m = 0;
#pragma unroll
  for (int t = 0; t < 16; ++t) m = pkmax(m, r[t] & 0x7fff7fffu);
  const u32 lo = lo_from_max(wave_max(max(m & 0xffffu, m >> 16)));
  Packed pk;
  u32 ha[4], hb[4];
#pragma unroll
  for (int j = 0; j < 4; ++j) {
    const u32 t0 = perm(r[4 * j + 1], r[4 * j], 0x05010400u), t1 = perm(r[4 * j + 1], r[4 * j], 0x07030602u);
    const u32 t2 = perm(r[4 * j + 3], r[4 * j + 2], 0x05010400u), t3 = perm(r[4 * j + 3], r[4 * j + 2], 0x07030602u);
    pk.la[j] = perm(t2, t0, 0x05040100u);
    ha[j] = perm(t2, t0, 0x07060302u);
    pk.lb[j] = perm(t3, t1, 0x05040100u);
    hb[j] = perm(t3, t1, 0x07060302u);
  }
  pack_hi(ha, hb, lo, true, pk);
  const int jd = cgl >> 3, cd = cgl & 7;
  const u32 fl = OFF_WF + (rank * 8 + cd) * 128 + (u8 >> 2) * 64 + (u8 & 3) * 8;
  send_unit<2>(P, pk, lo, jd, 4, OFF_WD + (u32)((rank * 8 + cd) * 8 + u8) * UNIT_B, fl, fl + 32, call, lane, tstart, dly);
}

__device__ __forceinline__ void decode_unit(const u32x4& la, const u32x4& lb, const u32x4& n, const u32x4& ov, u32 info,
                                            u64 badmask, u32 (&o)[16]) {
  const u32 lo4 = (info & 0x7f) * 0x01010101u;
  u32 ha[4], hb[4];
  unpack_hi(n, lo4, ha, hb);
  if (info & 0x80) {
    const bool bad = __builtin_amdgcn_inverse_ballot_w64(badmask);
#pragma unroll
    for (int j = 0; j < 4; ++j) {
      ha[j] = bad ? n[j] : ha[j];
      hb[j] = bad ? ov[j] : hb[j];
    }
  }
#pragma unroll
  for (int j = 0; j < 4; ++j) {
    o[2 * j] = perm(ha[j], la[j], 0x05010400u);
    o[2 * j + 1] = perm(ha[j], la[j], 0x07030602u);
    o[8 + 2 * j] = perm(hb[j], lb[j], 0x05010400u);
    o[8 + 2 * j + 1] = perm(hb[j], lb[j], 0x07030602u);
  }
}
__device__ __forceinline__ u64 rdlane64(const u32x2& v, int l) {
  return (u64)(u32)__builtin_amdgcn_readlane(v[0], l) | ((u64)(u32)__builtin_amdgcn_readlane(v[1], l) << 32);
}

__global__ __launch_bounds__(NT, 2) void gemm_ar_kernel(Params P) {
  extern __shared__ __attribute__((aligned(16))) char smem[];
  const int tid = threadIdx.x;
  const int lane = tid & 63;
  const int wave = __builtin_amdgcn_readfirstlane(tid >> 6);
  const int b = blockIdx.x;
  const int rank = P.rank;
  const int xcd = b & 7, idx = b >> 3, cls = idx >> 3, g = cls * 8 + xcd, c = idx & 7;
  char* myws = P.ws[rank];
  const int h = lane >> 5, l31 = lane & 31;
  const u64 tstart = __builtin_amdgcn_s_memrealtime();
  STAMP(0);

  const u32 call = __builtin_amdgcn_readfirstlane(
      __hip_atomic_load((gu32*)(myws + OFF_CNT + b * 128), __ATOMIC_RELAXED, __HIP_MEMORY_SCOPE_AGENT)) + 1;
  bool ok = true;

  constexpr u32 tw0 = TW0, dw = DW, tx0 = TX0, dx = DX, to0 = TO0, dO = DO;
  constexpr int gwfirst = GWFIRST;
  constexpr u32 lw = LW, lx = LX, lo_ = LO;
  if (wave == ((rank + 1) & 7)) send_w(P, b, call, lane, tstart, tw0 + ((b >> 3) & 7) * dw);
  if (wave == rank && g < gwfirst) send_x(P, g, c, call, lane, tstart, tx0 + g * dx);
  if (tid >= 256) {
    const u32 m = tid - 256;
    u32 sel[2] = {0u, 0u};
    u32 cnt = 0;
#pragma unroll
    for (int j = 0; j < 8; ++j) {
      const u32 bit = (m >> j) & 1;
      const u32 s = bit ? cnt : 0x0cu;
      sel[j >> 2] |= s << (8 * (j & 3));
      cnt += bit;
    }
    *(u32x2*)(smem + LDS_LUT + m * 8) = u32x2{sel[0], sel[1]};
  }
  STAMP(1);

  __amdgpu_buffer_rsrc_t rsme = mk_rsrc(myws, WS_NEED);
  u32x4 wf[2][16];
  {
    u64 fv;
    const u32 fl = OFF_WF + (wave * 8 + c) * 128;
    ok = wait_flags(myws, fl + (lane >> 2) * 64 + (lane & 3) * 8, lane < 8, call, lane, fv, tstart, max(tw0 + c * dw, 100u) + lw) && ok;
    STAMP(2);
    const u32 off = OFF_WD + (u32)((wave * 8 + c) * 8) * UNIT_B;
    const u32 finfo = (u32)(fv >> 32);
    const u32 ovm = (u32)__builtin_amdgcn_ballot_w64(lane < 8 && (finfo & 0x80) != 0);
    u32x4 pa[8], pb[8], pn[8], po[8];
    u32x2 mk = {0u, 0u};
    if (ovm != 0) {
      mk = __builtin_amdgcn_raw_buffer_load_b64(rsme, fl + ((lane >> 2) & 1) * 64 + 32 + (lane & 3) * 8, 0, AUX_SYS);
#pragma unroll
      for (int u8 = 0; u8 < 8; ++u8) {
        po[u8] = u32x4{0u, 0u, 0u, 0u};
        if ((ovm >> u8) & 1) po[u8] = __builtin_amdgcn_raw_buffer_load_b128(rsme, off + u8 * UNIT_B + 3072 + lane * 16, 0, AUX_SYS);
      }
    } else {
#pragma unroll
      for (int u8 = 0; u8 < 8; ++u8) po[u8] = u32x4{0u, 0u, 0u, 0u};
    }
#pragma unroll
    for (int u8 = 0; u8 < 8; ++u8) {
      pa[u8] = __builtin_amdgcn_raw_buffer_load_b128(rsme, off + u8 * UNIT_B + lane * 16, 0, AUX_SYS);
      pb[u8] = __builtin_amdgcn_raw_buffer_load_b128(rsme, off + u8 * UNIT_B + 1024 + lane * 16, 0, AUX_SYS);
      pn[u8] = __builtin_amdgcn_raw_buffer_load_b128(rsme, off + u8 * UNIT_B + 2048 + lane * 16, 0, AUX_SYS);
    }
    __builtin_amdgcn_sched_barrier(0);
#pragma unroll
    for (int u8 = 0; u8 < 8; ++u8) {
      const u32 info = __builtin_amdgcn_readlane(finfo, u8);
      u32 o[16];
      decode_unit(pa[u8], pb[u8], pn[u8], po[u8], info, rdlane64(mk, u8), o);
#pragma unroll
      for (int e = 0; e < 2; ++e)
#pragma unroll
        for (int q = 0; q < 4; ++q) {
          wf[0][2 * u8 + e][q] = o[4 * e + q];
          wf[1][2 * u8 + e][q] = o[8 + 4 * e + q];
        }
    }
  }
  STAMP(3);
  if (wave == rank && g >= gwfirst) send_x(P, g, c, call, lane, tstart, tx0 + g * dx);
  f32x16 acc0 = {}, acc1 = {};
  u32 tmv;
  {
    u64 fv;
    const u32 fl = OFF_XF + (wave * 32 + g) * 128;
    ok = wait_flags(myws, fl + lane * 8, lane < 4, call, lane, fv, tstart, tx0 + g * dx + lx) && ok;
    STAMP(4);
    tmv = read_tmo(myws);
    const u32 off = OFF_XD + (u32)((wave * 32 + g) * 4) * UNIT_B;
    const u32 finfo = (u32)(fv >> 32);
    const u32 ovm = (u32)__builtin_amdgcn_ballot_w64(lane < 4 && (finfo & 0x80) != 0);
    u32x4 pa[4], pb[4], pn[4], po[4];
    u32x2 mk = {0u, 0u};
    if (ovm != 0) {
      mk = __builtin_amdgcn_raw_buffer_load_b64(rsme, fl + 32 + (lane & 3) * 8, 0, AUX_SYS);
#pragma unroll
      for (int u = 0; u < 4; ++u) {
        po[u] = u32x4{0u, 0u, 0u, 0u};
        if ((ovm >> u) & 1) po[u] = __builtin_amdgcn_raw_buffer_load_b128(rsme, off + u * UNIT_B + 3072 + lane * 16, 0, AUX_SYS);
      }
    } else {
#pragma unroll
      for (int u = 0; u < 4; ++u) po[u] = u32x4{0u, 0u, 0u, 0u};
    }
#pragma unroll
    for (int u = 0; u < 4; ++u) {
      pa[u] = __builtin_amdgcn_raw_buffer_load_b128(rsme, off + u * UNIT_B + lane * 16, 0, AUX_SYS);
      pb[u] = __builtin_amdgcn_raw_buffer_load_b128(rsme, off + u * UNIT_B + 1024 + lane * 16, 0, AUX_SYS);
      pn[u] = __builtin_amdgcn_raw_buffer_load_b128(rsme, off + u * UNIT_B + 2048 + lane * 16, 0, AUX_SYS);
    }
    __builtin_amdgcn_sched_barrier(0);
#pragma unroll
    for (int u = 0; u < 4; ++u) {
      const u32 info = __builtin_amdgcn_readlane(finfo, u);
      u32 o[16];
      decode_unit(pa[u], pb[u], pn[u], po[u], info, rdlane64(mk, u), o);
#pragma unroll
      for (int e = 0; e < 4; ++e) {
        const u32x4 xf = {o[4 * e], o[4 * e + 1], o[4 * e + 2], o[4 * e + 3]};
        const int m = 4 * u + e;
        acc0 = __builtin_amdgcn_mfma_f32_32x32x16_bf16(__builtin_bit_cast(bf16x8, xf), __builtin_bit_cast(bf16x8, wf[0][m]), acc0, 0, 0, 0);
        acc1 = __builtin_amdgcn_mfma_f32_32x32x16_bf16(__builtin_bit_cast(bf16x8, xf), __builtin_bit_cast(bf16x8, wf[1][m]), acc1, 0, 0, 0);
      }
    }
  }
  STAMP(5);
  {
    char* red = smem + LDS_RED + wave * 8192;
#pragma unroll
    for (int r = 0; r < 16; ++r) {
      const int row = (r & 3) + 8 * (r >> 2) + 4 * h;
      const int off = (row * 256 + l31 * 8) ^ (((row >> 3) & 1) << 7);
      *(f32x2*)(red + off) = f32x2{acc0[r], acc1[r]};
    }
    const bool bad = !ok || __builtin_amdgcn_readfirstlane(tmv) == call;
    if (lane == 0) ((u32*)(smem + LDS_BAD))[wave] = bad ? 1u : 0u;
  }
  __syncthreads();
  u32 o0, o1, nib, lpre;
  {
    const int cg = lane >> 3, q = (lane >> 1) & 3, hf = lane & 1;
    const int row = 8 * q + wave;
    const int off = (row * 256 + cg * 32 + hf * 16) ^ ((q & 1) << 7);
    f32x4 s = *(const f32x4*)(smem + LDS_RED + off);
#pragma unroll
    for (int w2 = 1; w2 < 8; ++w2) s += *(const f32x4*)(smem + LDS_RED + w2 * 8192 + off);
    const u32x4 bw0 = *(const u32x4*)(smem + LDS_BAD), bw1 = *(const u32x4*)(smem + LDS_BAD + 16);
    const bool bad = (bw0[0] | bw0[1] | bw0[2] | bw0[3] | bw1[0] | bw1[1] | bw1[2] | bw1[3]) != 0;
    float v[4];
#pragma unroll
    for (int e = 0; e < 4; ++e) v[e] = !(s[e] <= 0.f) ? s[e] : 0.f;
    o0 = cvt_pk(v[0], v[1]);
    o1 = cvt_pk(v[2], v[3]);
    if (bad) { o0 = 0x7fc07fc0u; o1 = 0x7fc07fc0u; }
    nib = ((o0 & 0xffffu) ? 1u : 0u) | ((o0 >> 16) ? 2u : 0u) | ((o1 & 0xffffu) ? 4u : 0u) | ((o1 >> 16) ? 8u : 0u);
    const u32 cnt = __builtin_popcount(nib);
    const u32 incl = wave_scan_incl(cnt);
    lpre = incl - cnt;
    const u32 tot = __builtin_amdgcn_readlane(incl, 63);
    const u32 pm = pkmax(o0 & 0x7fff7fffu, o1 & 0x7fff7fffu);
    const u32 mx = wave_max(max(pm & 0xffffu, pm >> 16));
    if (lane == 0) ((u32*)(smem + LDS_INFO))[wave] = tot | (mx << 16);
  }
  __syncthreads();
  u32 count, mx16;
  {
    const u32x4 i0 = *(const u32x4*)(smem + LDS_INFO), i1 = *(const u32x4*)(smem + LDS_INFO + 16);
    u32 pre = 0, tot = 0, mx = 0;
#pragma unroll
    for (int w2 = 0; w2 < 8; ++w2) {
      const u32 iv = w2 < 4 ? i0[w2 & 3] : i1[w2 & 3];
      if (w2 == wave) pre = tot;
      tot += iv & 0xffffu;
      mx = max(mx, iv >> 16);
    }
    count = __builtin_amdgcn_readfirstlane(tot);
    mx16 = __builtin_amdgcn_readfirstlane(mx);
    uint16_t* str = (uint16_t*)(smem + LDS_STR);
    u32 p = pre + lpre;
    if (nib & 1) str[p] = (uint16_t)o0;
    p += nib & 1;
    if (nib & 2) str[p] = (uint16_t)(o0 >> 16);
    p += (nib >> 1) & 1;
    if (nib & 4) str[p] = (uint16_t)o1;
    p += (nib >> 2) & 1;
    if (nib & 8) str[p] = (uint16_t)(o1 >> 16);
    const u32 nnext = (u32)__builtin_amdgcn_update_dpp(0, (int)nib, 0x101  , 0xf, 0xf, false);
    if ((lane & 1) == 0) *(uint8_t*)(smem + LDS_MSK + wave * 32 + (lane >> 1)) = (uint8_t)(nib | (nnext << 4));
    if (wave < 2) str[count + wave * 64 + lane] = (uint16_t)mx16;
  }
  __syncthreads();
  STAMP(6);
  {
    const u32 nv = ((count + 127) >> 7) << 2;
    const u32 lo = lo_from_max(mx16);
    const bool on = (u32)lane < nv;
    const char* sa = smem + LDS_STR + lane * 32;
    const char* sb = sa + nv * 32;
    u32 r[16];
    {
      const u32x4 t0 = *(const u32x4*)(sa), t1 = *(const u32x4*)(sa + 16);
      const u32x4 t2 = *(const u32x4*)(on ? sb : sa), t3 = *(const u32x4*)((on ? sb : sa) + 16);
#pragma unroll
      for (int q = 0; q < 4; ++q) { r[q] = t0[q]; r[4 + q] = t1[q]; r[8 + q] = t2[q]; r[12 + q] = t3[q]; }
    }
    Packed pk;
    u32 ha[4], hb[4];
#pragma unroll
    for (int j = 0; j < 4; ++j) {
      pk.la[j] = perm(r[2 * j + 1], r[2 * j], 0x06040200u);
      ha[j] = perm(r[2 * j + 1], r[2 * j], 0x07050301u);
      pk.lb[j] = perm(r[8 + 2 * j + 1], r[8 + 2 * j], 0x06040200u);
      hb[j] = perm(r[8 + 2 * j + 1], r[8 + 2 * j], 0x07050301u);
    }
    pack_hi(ha, hb, lo, on, pk);
    __amdgpu_buffer_rsrc_t rs = mk_rsrc(P.ws[wave], WS_NEED);
    const u32 slot = OFF_OD + (u32)(rank * 256 + g * 8 + c) * OT_B;
    timer_wait(tstart, to0 + g * dO);
    if (lane < 16) {
      const u32x4 mk = *(const u32x4*)(smem + LDS_MSK + lane * 16);
      __builtin_amdgcn_raw_buffer_store_b128(mk, rs, slot + lane * 16, 0, AUX_SYS);
    }
    if (on) {
      __builtin_amdgcn_raw_buffer_store_b128(pk.la, rs, slot + 256 + lane * 16, 0, AUX_SYS);
      __builtin_amdgcn_raw_buffer_store_b128(pk.lb, rs, slot + 256 + (nv + lane) * 16, 0, AUX_SYS);
      __builtin_amdgcn_raw_buffer_store_b128(pk.n, rs, slot + 256 + (2 * nv + lane) * 16, 0, AUX_SYS);
    }
    if (pk.bad != 0) {
      if ((pk.bad >> lane) & 1) __builtin_amdgcn_raw_buffer_store_b128(pk.ovf, rs, slot + OT_OVF + lane * 16, 0, AUX_SYS);
      if (lane == 0) __builtin_amdgcn_raw_buffer_store_b64(u32x2{(u32)pk.bad, (u32)(pk.bad >> 32)}, rs, slot + OT_BM, 0, AUX_SYS);
    }
    asm volatile("s_waitcnt vmcnt(0)" ::: "memory");
    if (lane == 0) {
      const u32x2 f = {call, count | (lo << 12) | (pk.bad != 0 ? 1u << 19 : 0u)};
      __builtin_amdgcn_raw_buffer_store_b64(f, rs, OFF_OF + (rank * 256 + g * 8 + c) * 128, 0, AUX_SYS);
    }
  }
  STAMP(7);
  {
    u64 fv;
    const bool ok2 = wait_flags(myws, OFF_OF + (wave * 256 + g * 8 + c) * 128, lane == 0, call, lane, fv, tstart, to0 + g * dO + lo_);
    STAMP(8);
    const u32 tm2 = read_tmo(myws);
    const u32 finfo = __builtin_amdgcn_readlane((u32)(fv >> 32), 0);
    const u32 cnt2 = ok2 ? (finfo & 0xfffu) : 0u;
    const u32 lo4 = ((finfo >> 12) & 0x7f) * 0x01010101u;
    const u32 nv = ((cnt2 + 127) >> 7) << 2;
    const u32 slot = OFF_OD + (u32)(wave * 256 + g * 8 + c) * OT_B;
    const u32 m = __builtin_amdgcn_raw_buffer_load_b32(rsme, slot + lane * 4, 0, AUX_SYS);
    char* ua = smem + LDS_RED + wave * UAREA;
    u32x4 ov = {0u, 0u, 0u, 0u};
    u32x2 mk = {0u, 0u};
    if (finfo & (1u << 19)) {
      ov = __builtin_amdgcn_raw_buffer_load_b128(rsme, slot + OT_OVF + lane * 16, 0, AUX_SYS);
      mk = __builtin_amdgcn_raw_buffer_load_b64(rsme, slot + OT_BM, 0, AUX_SYS);
    }
    if ((u32)lane < nv) {
      const u32x4 la = __builtin_amdgcn_raw_buffer_load_b128(rsme, slot + 256 + lane * 16, 0, AUX_SYS);
      const u32x4 lb = __builtin_amdgcn_raw_buffer_load_b128(rsme, slot + 256 + (nv + lane) * 16, 0, AUX_SYS);
      const u32x4 n = __builtin_amdgcn_raw_buffer_load_b128(rsme, slot + 256 + (2 * nv + lane) * 16, 0, AUX_SYS);
      u32 ha[4], hb[4];
      unpack_hi(n, lo4, ha, hb);
      if (finfo & (1u << 19)) {
        const bool bad = ((lane < 32 ? mk[0] >> lane : mk[1] >> (lane - 32)) & 1) != 0;
#pragma unroll
        for (int j = 0; j < 4; ++j) {
          ha[j] = bad ? n[j] : ha[j];
          hb[j] = bad ? ov[j] : hb[j];
        }
      }
      *(u32x4*)(ua + lane * 16) = la;
      *(u32x4*)(ua + (nv + lane) * 16) = lb;
      *(u32x4*)(ua + 2176 + lane * 16) = u32x4{ha[0], ha[1], ha[2], ha[3]};
      *(u32x4*)(ua + 2176 + (nv + lane) * 16) = u32x4{hb[0], hb[1], hb[2], hb[3]};
    }
    const bool bad = !ok2 || __builtin_amdgcn_readfirstlane(tm2) == call;
    const u32 pc = __builtin_popcount(m);
    const u32 pos0 = wave_scan_incl(pc) - pc;
    const int si = wave >> 2, sj = wave & 3;
    const u32 obase = ((u32)(1024 * si + 32 * g + (lane >> 3)) * NN + 512 * sj + 64 * c + 8 * (lane & 7)) * 2;
    __amdgpu_buffer_rsrc_t rso = mk_rsrc(P.out, MM * NN * 2);
    asm volatile("" ::: "memory");
    __builtin_amdgcn_wave_barrier();
#pragma unroll
    for (int q = 0; q < 4; ++q) {
      const u32 mq = (m >> (8 * q)) & 0xff;
      u32 pos = pos0 + __builtin_popcount(q == 0 ? 0u : (m & ((1u << (8 * q)) - 1u)));
      pos = min(pos, 2048u);
      const u32x2 sel = *(const u32x2*)(smem + LDS_LUT + mq * 8);
      const u32* lw = (const u32*)(ua + (pos & ~3u));
      const u32* hw = (const u32*)(ua + 2176 + (pos & ~3u));
      const u32 l0 = lw[0], l1 = lw[1], l2 = lw[2];
      const u32 h0 = hw[0], h1 = hw[1], h2 = hw[2];
      const u32 la0 = __builtin_amdgcn_alignbyte(l1, l0, pos), la1 = __builtin_amdgcn_alignbyte(l2, l1, pos);
      const u32 ha0 = __builtin_amdgcn_alignbyte(h1, h0, pos), ha1 = __builtin_amdgcn_alignbyte(h2, h1, pos);
      const u32 eL0 = perm(la1, la0, sel[0]), eL1 = perm(la1, la0, sel[1]);
      const u32 eH0 = perm(ha1, ha0, sel[0]), eH1 = perm(ha1, ha0, sel[1]);
      u32x4 o = {perm(eH0, eL0, 0x05010400u), perm(eH0, eL0, 0x07030602u), perm(eH1, eL1, 0x05010400u), perm(eH1, eL1, 0x07030602u)};
      if (bad) o = u32x4{0x7fc07fc0u, 0x7fc07fc0u, 0x7fc07fc0u, 0x7fc07fc0u};
      __builtin_amdgcn_raw_buffer_store_b128(o, rso, obase + q * 8 * NN * 2, 0, OUT_AUX);
    }
  }
  if (tid == 0) __hip_atomic_store((gu32*)(myws + OFF_CNT + b * 128), call, __ATOMIC_RELAXED, __HIP_MEMORY_SCOPE_AGENT);
  STAMP(9);
}

}

extern "C" void kernel_launch(int rank, int nranks, ncclComm_t comm, hipStream_t stream,
                              void* const* d_in, const int* in_sizes, int n_in,
                              void* d_out, int out_size, const int* in_dtypes, int out_dtype,
                              void* const* d_out_all, void* const* d_ws_all, size_t ws_bytes) {
  if (nranks != NR || ws_bytes < WS_NEED || (unsigned)rank >= (unsigned)NR) {
    fprintf(stderr, "unexpected configuration\n");
    return;
  }
  static bool init[NR] = {};
  if (!init[rank]) {
    HIP_CHECK(hipFuncSetAttribute((const void*)gemm_ar_kernel, hipFuncAttributeMaxDynamicSharedMemorySize, LDS_BYTES));
    init[rank] = true;
  }
  Params p;
  p.x = (const uint16_t*)d_in[0];
  p.w = (const uint16_t*)d_in[1];
  for (int r = 0; r < NR; ++r) p.ws[r] = (char*)d_ws_all[r];
  p.out = (char*)d_out;
  p.rank = rank;
  p.pad = 0;
  hipLaunchKernelGGL(gemm_ar_kernel, dim3(NWG), dim3(NT), LDS_BYTES, stream, p);
}
```

```cpp
#include <hip/hip_runtime.h>
#include <rccl/rccl.h>
#include <cstdio>
#include <cstdint>
#include <cstdlib>

#define HIP_CHECK(expr) do { hipError_t _e = (expr); if (_e != hipSuccess) { \
    fprintf(stderr, "HIP error %s at %s:%d\n", hipGetErrorString(_e), __FILE__, __LINE__); \
    return; } } while (0)

namespace {

typedef unsigned u32;
typedef unsigned long long u64;
typedef u32 u32x4 __attribute__((ext_vector_type(4)));
typedef u32 u32x2 __attribute__((ext_vector_type(2)));
typedef float f32x16 __attribute__((ext_vector_type(16)));
typedef float f32x4 __attribute__((ext_vector_type(4)));
typedef float f32x2 __attribute__((ext_vector_type(2)));
typedef __bf16 bf16x8 __attribute__((ext_vector_type(8)));
typedef __bf16 bf16x2 __attribute__((ext_vector_type(2)));
typedef unsigned short u16x2 __attribute__((ext_vector_type(2)));
typedef __attribute__((address_space(1))) u64 gu64;
typedef __attribute__((address_space(1))) u32 gu32;

constexpr int NR = 8;
constexpr int MM = 2048, NN = 2048, KL = 256;
constexpr int NWG = 256, NT = 512;

constexpr u32 OFF_CNT = 0;
constexpr u32 OFF_TMO = OFF_CNT + NWG * 128;
constexpr u32 OFF_WF = 65536;
constexpr u32 OFF_XF = OFF_WF + 64 * 128;
constexpr u32 OFF_OF = OFF_XF + 256 * 128;
constexpr u32 OFF_DBG = OFF_OF + 2048 * 128;
constexpr u32 UNIT_B = 4096;
constexpr u32 OFF_WD = 1u << 20;
constexpr u32 OFF_XD = OFF_WD + 8 * 8 * 8 * UNIT_B;
constexpr u32 OT_B = 4608;
constexpr u32 OT_OVF = 3328, OT_BM = 4352;
constexpr u32 OFF_OD = OFF_XD + 8 * 32 * 4 * UNIT_B;
constexpr u32 WS_NEED = OFF_OD + 2048 * OT_B;
static_assert(OFF_DBG + 256 * 8 * 16 * 8 <= OFF_WD, "layout");

constexpr u64 DEADLINE = 20000000ull;
constexpr int AUX_SYS = 17;

#define TW0 130
#define DW 45
#define TX0 480
#define DX 25
#define TO0 1200
#define DO 30
#define LW 150
#define LX 150
#define LO 150
#define PSLEEP 4
#define OUT_AUX 16
#define MAPB 1
#define GWFIRST 14

constexpr int LDS_RED = 0;
constexpr int UAREA = 4352;
constexpr int LDS_STR = 65536;
constexpr int LDS_MSK = LDS_STR + 4352;
constexpr int LDS_INFO = LDS_MSK + 256;
constexpr int LDS_BAD = LDS_INFO + 32;
constexpr int LDS_LUT = LDS_BAD + 32;
constexpr int LDS_BYTES = LDS_LUT + 2048;
static_assert(8 * UAREA <= 65536, "uarea");

struct Params {
  const uint16_t* x;
  const uint16_t* w;
  char* ws[NR];
  char* out;
  int rank;
  int pad;
};

__device__ __forceinline__ __amdgpu_buffer_rsrc_t mk_rsrc(const void* p, u32 bytes) {
  return __builtin_amdgcn_make_buffer_rsrc((void*)p, 0, bytes, 0x00020000);
}
__device__ __forceinline__ u32 perm(u32 hi, u32 lo, u32 sel) { return __builtin_amdgcn_perm(hi, lo, sel); }
__device__ __forceinline__ u32 rotr4(u32 v) { return __builtin_amdgcn_alignbit(v, v, 4); }
__device__ __forceinline__ u32 rotl4(u32 v) { return __builtin_amdgcn_alignbit(v, v, 28); }
__device__ __forceinline__ u32 pkmax(u32 a, u32 b) {
  return __builtin_bit_cast(u32, __builtin_elementwise_max(__builtin_bit_cast(u16x2, a), __builtin_bit_cast(u16x2, b)));
}
template <int CTRL, int RM>
__device__ __forceinline__ u32 dpp0(u32 v) { return (u32)__builtin_amdgcn_update_dpp(0, (int)v, CTRL, RM, 0xf, false); }
__device__ __forceinline__ u32 wave_max(u32 v) {
  v = max(v, dpp0<0x111, 0xf>(v));
  v = max(v, dpp0<0x112, 0xf>(v));
  v = max(v, dpp0<0x114, 0xf>(v));
  v = max(v, dpp0<0x118, 0xf>(v));
  v = max(v, dpp0<0x142, 0xa>(v));
  v = max(v, dpp0<0x143, 0xc>(v));
  return (u32)__builtin_amdgcn_readlane((int)v, 63);
}
__device__ __forceinline__ u32 wave_scan_incl(u32 v) {
  v += dpp0<0x111, 0xf>(v);
  v += dpp0<0x112, 0xf>(v);
  v += dpp0<0x114, 0xf>(v);
  v += dpp0<0x118, 0xf>(v);
  v += dpp0<0x142, 0xa>(v);
  v += dpp0<0x143, 0xc>(v);
  return v;
}

#define STAMP(i) do {} while (0)
#define STAMP2(i) do {} while (0)

__device__ __forceinline__ bool wait_flags(char* myws, u32 off, bool act, u32 call, int lane, u64& val, u64 tstart, u32 notbefore) {
  gu64* addr = (gu64*)(myws + off);
  while (__builtin_amdgcn_s_memrealtime() - tstart < (u64)notbefore) __builtin_amdgcn_s_sleep(2);
  const u64 t0 = __builtin_amdgcn_s_memrealtime();
  bool ok = true;
  for (;;) {
    u64 v = (u64)call;
    if (act) v = __hip_atomic_load(addr, __ATOMIC_RELAXED, __HIP_MEMORY_SCOPE_SYSTEM);
    val = v;
    const u64 pend = __builtin_amdgcn_ballot_w64(act && (u32)v < call);
    if (pend == 0) break;
    const u64 el = __builtin_amdgcn_s_memrealtime() - t0;
    if (el > 50000) {
      const u32 tm = (u32)__hip_atomic_load((gu64*)(myws + OFF_TMO), __ATOMIC_RELAXED, __HIP_MEMORY_SCOPE_AGENT);
      if (__builtin_amdgcn_readfirstlane(tm) == call || el > DEADLINE) {
        if (lane == 0) __hip_atomic_store((gu64*)(myws + OFF_TMO), (u64)call, __ATOMIC_RELAXED, __HIP_MEMORY_SCOPE_AGENT);
        ok = false;
        break;
      }
    }
    __builtin_amdgcn_s_sleep(PSLEEP);
  }
  asm volatile("" ::: "memory");
  return ok;
}
__device__ __forceinline__ u32 read_tmo(char* myws) {
  return (u32)__hip_atomic_load((gu64*)(myws + OFF_TMO), __ATOMIC_RELAXED, __HIP_MEMORY_SCOPE_AGENT);
}

__device__ __forceinline__ u32 cvt_pk(float a, float b_) {
  f32x2 f = {a, b_};
  return __builtin_bit_cast(u32, __builtin_convertvector(f, bf16x2));
}

struct Packed { u32x4 la, lb, n, ovf; u64 bad; };

__device__ __forceinline__ void pack_hi(const u32 (&ha)[4], const u32 (&hb)[4], u32 lo, bool lane_on, Packed& pk) {
  const u32 lo4 = lo * 0x01010101u;
  u32 acc = 0x80808080u;
#pragma unroll
  for (int j = 0; j < 4; ++j) {
    const u32 da = (ha[j] | 0x80808080u) - lo4;
    const u32 db = (hb[j] | 0x80808080u) - lo4;
    acc &= da & db;
    const u32 xa = (ha[j] & 0x80808080u) | (da & 0x7f7f7f7fu);
    const u32 xb = (hb[j] & 0x80808080u) | (db & 0x7f7f7f7fu);
    pk.n[j] = xa | rotr4(xb);
  }
  const bool bad = lane_on && acc != 0x80808080u;
  pk.bad = __builtin_amdgcn_ballot_w64(bad);
  if (pk.bad != 0) {
#pragma unroll
    for (int j = 0; j < 4; ++j) {
      pk.n[j] = bad ? ha[j] : pk.n[j];
      pk.ovf[j] = hb[j];
    }
  }
}
__device__ __forceinline__ void unpack_hi(const u32x4& n, u32 lo4, u32 (&ha)[4], u32 (&hb)[4]) {
#pragma unroll
  for (int j = 0; j < 4; ++j) {
    ha[j] = (n[j] & 0x87878787u) + lo4;
    hb[j] = (rotl4(n[j]) & 0x87878787u) + lo4;
  }
}
__device__ __forceinline__ u32 lo_from_max(u32 m16) {
  const u32 e7 = m16 >> 8;
  return e7 > 7 ? e7 - 7 : 0;
}

__device__ __forceinline__ void timer_wait(u64 tstart, u32 dly) {
  while (__builtin_amdgcn_s_memrealtime() - tstart < (u64)dly) __builtin_amdgcn_s_sleep(1);
}

template <int ND>
__device__ __forceinline__ void send_unit(const Params& P, const Packed& pk, u32 lo, int dst0, int dstep, u32 doff,
                                          u32 foff, u32 moff, u32 call, int lane, u64 tstart, u32 dly) {
  STAMP2(10 + (ND == 4 ? 2 : 0));
  timer_wait(tstart, dly);
  STAMP2(14 + (ND == 4 ? 1 : 0));
#pragma unroll
  for (int d = 0; d < ND; ++d) {
    if (dst0 + d * dstep == P.rank) continue;
    __amdgpu_buffer_rsrc_t rs = mk_rsrc(P.ws[dst0 + d * dstep], WS_NEED);
    __builtin_amdgcn_raw_buffer_store_b128(pk.la, rs, doff + lane * 16, 0, AUX_SYS);
    __builtin_amdgcn_raw_buffer_store_b128(pk.lb, rs, doff + 1024 + lane * 16, 0, AUX_SYS);
    __builtin_amdgcn_raw_buffer_store_b128(pk.n, rs, doff + 2048 + lane * 16, 0, AUX_SYS);
  }
  if (pk.bad != 0) {
#pragma unroll
    for (int d = 0; d < ND; ++d) {
      if (dst0 + d * dstep == P.rank) continue;
      __amdgpu_buffer_rsrc_t rs = mk_rsrc(P.ws[dst0 + d * dstep], WS_NEED);
      if ((pk.bad >> lane) & 1) __builtin_amdgcn_raw_buffer_store_b128(pk.ovf, rs, doff + 3072 + lane * 16, 0, AUX_SYS);
      if (lane == 0) __builtin_amdgcn_raw_buffer_store_b64(u32x2{(u32)pk.bad, (u32)(pk.bad >> 32)}, rs, moff, 0, AUX_SYS);
    }
  }
  asm volatile("s_waitcnt vmcnt(0)" ::: "memory");
  STAMP2(11 + (ND == 4 ? 2 : 0));
  const u32x2 f = {call, lo | (pk.bad != 0 ? 0x80u : 0u)};
  if (lane == 0) {
#pragma unroll
    for (int d = 0; d < ND; ++d)
      if (dst0 + d * dstep != P.rank)
        __builtin_amdgcn_raw_buffer_store_b64(f, mk_rsrc(P.ws[dst0 + d * dstep], WS_NEED), foff, 0, AUX_SYS);
  }
}

__device__ __forceinline__ void send_x(const Params& P, int g, int c, u32 call, int lane, u64 tstart, u32 dly) {
  const int rank = P.rank, h = lane >> 5, l31 = lane & 31;
  const int ih = c >> 2, u = c & 3;
  const u32x4* src = (const u32x4*)(P.x + (size_t)(1024 * ih + 32 * g + l31) * KL + 128 * h + 32 * u);
  u32 r[16];
#pragma unroll
  for (int p = 0; p < 4; ++p) {
    const u32x4 t = src[p];
#pragma unroll
    for (int q = 0; q < 4; ++q) r[4 * p + q] = t[q];
  }
  u32 m = 0;
#pragma unroll
  for (int t = 0; t < 16; ++t) m = pkmax(m, r[t] & 0x7fff7fffu);
  const u32 lo = lo_from_max(wave_max(max(m & 0xffffu, m >> 16)));
  Packed pk;
  u32 ha[4], hb[4];
#pragma unroll
  for (int j = 0; j < 4; ++j) {
    pk.la[j] = perm(r[2 * j + 1], r[2 * j], 0x06040200u);
    ha[j] = perm(r[2 * j + 1], r[2 * j], 0x07050301u);
    pk.lb[j] = perm(r[8 + 2 * j + 1], r[8 + 2 * j], 0x06040200u);
    hb[j] = perm(r[8 + 2 * j + 1], r[8 + 2 * j], 0x07050301u);
  }
  pack_hi(ha, hb, lo, true, pk);
  const u32 fl = OFF_XF + (rank * 32 + g) * 128;
  send_unit<4>(P, pk, lo, 4 * ih, 1, OFF_XD + (u32)((rank * 32 + g) * 4 + u) * UNIT_B, fl + u * 8, fl + 32 + u * 8, call, lane, tstart, dly);
}

__device__ __forceinline__ void send_w(const Params& P, int b, u32 call, int lane, u64 tstart, u32 dly) {
  const int rank = P.rank, h = lane >> 5, l31 = lane & 31;
  const int cgl = b >> 3, u8 = b & 7;
  const u32* src = (const u32*)(P.w + (size_t)(128 * h + 16 * u8) * NN + 64 * cgl + 2 * l31);
  u32 r[16];
#pragma unroll
  for (int k = 0; k < 16; ++k) r[k] = src[k * (NN / 2)];
  u32 m = 0;
#pragma unroll
  for (int t = 0; t < 16; ++t) m = pkmax(m, r[t] & 0x7fff7fffu);
  const u32 lo = lo_from_max(wave_max(max(m & 0xffffu, m >> 16)));
  Packed pk;
  u32 ha[4], hb[4];
#pragma unroll
  for (int j = 0; j < 4; ++j) {
    const u32 t0 = perm(r[4 * j + 1], r[4 * j], 0x05010400u), t1 = perm(r[4 * j + 1], r[4 * j], 0x07030602u);
    const u32 t2 = perm(r[4 * j + 3], r[4 * j + 2], 0x05010400u), t3 = perm(r[4 * j + 3], r[4 * j + 2], 0x07030602u);
    pk.la[j] = perm(t2, t0, 0x05040100u);
    ha[j] = perm(t2, t0, 0x07060302u);
    pk.lb[j] = perm(t3, t1, 0x05040100u);
    hb[j] = perm(t3, t1, 0x07060302u);
  }
  pack_hi(ha, hb, lo, true, pk);
  const int jd = cgl >> 3, cd = cgl & 7;
  const u32 fl = OFF_WF + (rank * 8 + cd) * 128 + (u8 >> 2) * 64 + (u8 & 3) * 8;
  send_unit<2>(P, pk, lo, jd, 4, OFF_WD + (u32)((rank * 8 + cd) * 8 + u8) * UNIT_B, fl, fl + 32, call, lane, tstart, dly);
}

__device__ __forceinline__ void decode_unit(const u32x4& la, const u32x4& lb, const u32x4& n, const u32x4& ov, u32 info,
                                            u64 badmask, u32 (&o)[16]) {
  const u32 lo4 = (info & 0x7f) * 0x01010101u;
  u32 ha[4], hb[4];
  unpack_hi(n, lo4, ha, hb);
  if (info & 0x80) {
    const bool bad = __builtin_amdgcn_inverse_ballot_w64(badmask);
#pragma unroll
    for (int j = 0; j < 4; ++j) {
      ha[j] = bad ? n[j] : ha[j];
      hb[j] = bad ? ov[j] : hb[j];
    }
  }
#pragma unroll
  for (int j = 0; j < 4; ++j) {
    o[2 * j] = perm(ha[j], la[j], 0x05010400u);
    o[2 * j + 1] = perm(ha[j], la[j], 0x07030602u);
    o[8 + 2 * j] = perm(hb[j], lb[j], 0x05010400u);
    o[8 + 2 * j + 1] = perm(hb[j], lb[j], 0x07030602u);
  }
}
__device__ __forceinline__ u64 rdlane64(const u32x2& v, int l) {
  return (u64)(u32)__builtin_amdgcn_readlane(v[0], l) | ((u64)(u32)__builtin_amdgcn_readlane(v[1], l) << 32);
}

__global__ __launch_bounds__(NT, 2) void gemm_ar_kernel(Params P) {
  extern __shared__ __attribute__((aligned(16))) char smem[];
  const int tid = threadIdx.x;
  const int lane = tid & 63;
  const int wave = __builtin_amdgcn_readfirstlane(tid >> 6);
  const int b = blockIdx.x;
  const int rank = P.rank;
  const int g = b >> 3, c = b & 7;
  char* myws = P.ws[rank];
  const int h = lane >> 5, l31 = lane & 31;
  const u64 tstart = __builtin_amdgcn_s_memrealtime();
  STAMP(0);

  const u32 call = __builtin_amdgcn_readfirstlane(
      __hip_atomic_load((gu32*)(myws + OFF_CNT + b * 128), __ATOMIC_RELAXED, __HIP_MEMORY_SCOPE_AGENT)) + 1;
  bool ok = true;

  constexpr u32 tw0 = TW0, dw = DW, tx0 = TX0, dx = DX, to0 = TO0, dO = DO;
  constexpr int gwfirst = GWFIRST;
  constexpr u32 lw = LW, lx = LX, lo_ = LO;
  __amdgpu_buffer_rsrc_t rsme = mk_rsrc(myws, WS_NEED);
  f32x16 acc0 = {}, acc1 = {};
  u32 tmv = 0;
  if (wave == rank) {
    u32x4 wf[2][16];
    {
      const u32* src = (const u32*)(P.w + (size_t)(128 * h) * NN + 512 * (rank & 3) + 64 * c + 2 * l31);
#pragma unroll
      for (int m2 = 0; m2 < 2; ++m2) {
        u32 r[64];
#pragma unroll
        for (int k = 0; k < 64; ++k) r[k] = src[(64 * m2 + k) * (NN / 2)];
        __builtin_amdgcn_sched_barrier(0);
#pragma unroll
        for (int e = 0; e < 8; ++e)
#pragma unroll
          for (int q = 0; q < 4; ++q) {
            wf[0][8 * m2 + e][q] = perm(r[8 * e + 2 * q + 1], r[8 * e + 2 * q], 0x05040100u);
            wf[1][8 * m2 + e][q] = perm(r[8 * e + 2 * q + 1], r[8 * e + 2 * q], 0x07060302u);
          }
      }
    }
    STAMP(3);
    {
      const u32x4* src = (const u32x4*)(P.x + (size_t)(1024 * (rank >> 2) + 32 * g + l31) * KL + 128 * h);
      u32x4 xf[16];
#pragma unroll
      for (int m = 0; m < 16; ++m) xf[m] = src[m];
      __builtin_amdgcn_sched_barrier(0);
#pragma unroll
      for (int m = 0; m < 16; ++m) {
        acc0 = __builtin_amdgcn_mfma_f32_32x32x16_bf16(__builtin_bit_cast(bf16x8, xf[m]), __builtin_bit_cast(bf16x8, wf[0][m]), acc0, 0, 0, 0);
        acc1 = __builtin_amdgcn_mfma_f32_32x32x16_bf16(__builtin_bit_cast(bf16x8, xf[m]), __builtin_bit_cast(bf16x8, wf[1][m]), acc1, 0, 0, 0);
      }
    }
    STAMP(4);
    send_x(P, g, c, call, lane, tstart, tx0 + g * dx);
    tmv = read_tmo(myws);
  } else {
    const u32 rel = (wave - rank - 1) & 7;
    if (rel == 0) send_w(P, b, call, lane, tstart, tw0 + ((b >> 3) & 7) * dw);
    const u32 lw4 = rel - 1;
    if (lw4 < 4) {
      const u32 m = lw4 * 64 + lane;
      u32 sel[2] = {0u, 0u};
      u32 cnt = 0;
#pragma unroll
      for (int j = 0; j < 8; ++j) {
        const u32 bit = (m >> j) & 1;
        const u32 s = bit ? cnt : 0x0cu;
        sel[j >> 2] |= s << (8 * (j & 3));
        cnt += bit;
      }
      *(u32x2*)(smem + LDS_LUT + m * 8) = u32x2{sel[0], sel[1]};
    }
    STAMP(1);
    u32x4 wf[2][16];
  {
    u64 fv;
    const u32 fl = OFF_WF + (wave * 8 + c) * 128;
    ok = wait_flags(myws, fl + (lane >> 2) * 64 + (lane & 3) * 8, lane < 8, call, lane, fv, tstart, max(tw0 + c * dw, 100u) + lw) && ok;
    STAMP(2);
    const u32 off = OFF_WD + (u32)((wave * 8 + c) * 8) * UNIT_B;
    const u32 finfo = (u32)(fv >> 32);
    const u32 ovm = (u32)__builtin_amdgcn_ballot_w64(lane < 8 && (finfo & 0x80) != 0);
    u32x4 pa[8], pb[8], pn[8], po[8];
    u32x2 mk = {0u, 0u};
    if (ovm != 0) {
      mk = __builtin_amdgcn_raw_buffer_load_b64(rsme, fl + ((lane >> 2) & 1) * 64 + 32 + (lane & 3) * 8, 0, AUX_SYS);
#pragma unroll
      for (int u8 = 0; u8 < 8; ++u8) {
        po[u8] = u32x4{0u, 0u, 0u, 0u};
        if ((ovm >> u8) & 1) po[u8] = __builtin_amdgcn_raw_buffer_load_b128(rsme, off + u8 * UNIT_B + 3072 + lane * 16, 0, AUX_SYS);
      }
    } else {
#pragma unroll
      for (int u8 = 0; u8 < 8; ++u8) po[u8] = u32x4{0u, 0u, 0u, 0u};
    }
#pragma unroll
    for (int u8 = 0; u8 < 8; ++u8) {
      pa[u8] = __builtin_amdgcn_raw_buffer_load_b128(rsme, off + u8 * UNIT_B + lane * 16, 0, AUX_SYS);
      pb[u8] = __builtin_amdgcn_raw_buffer_load_b128(rsme, off + u8 * UNIT_B + 1024 + lane * 16, 0, AUX_SYS);
      pn[u8] = __builtin_amdgcn_raw_buffer_load_b128(rsme, off + u8 * UNIT_B + 2048 + lane * 16, 0, AUX_SYS);
    }
    __builtin_amdgcn_sched_barrier(0);
#pragma unroll
    for (int u8 = 0; u8 < 8; ++u8) {
      const u32 info = __builtin_amdgcn_readlane(finfo, u8);
      u32 o[16];
      decode_unit(pa[u8], pb[u8], pn[u8], po[u8], info, rdlane64(mk, u8), o);
#pragma unroll
      for (int e = 0; e < 2; ++e)
#pragma unroll
        for (int q = 0; q < 4; ++q) {
          wf[0][2 * u8 + e][q] = o[4 * e + q];
          wf[1][2 * u8 + e][q] = o[8 + 4 * e + q];
        }
    }
  }
  STAMP(3);
  {
    u64 fv;
    const u32 fl = OFF_XF + (wave * 32 + g) * 128;
    ok = wait_flags(myws, fl + lane * 8, lane < 4, call, lane, fv, tstart, tx0 + g * dx + lx) && ok;
    STAMP(4);
    tmv = read_tmo(myws);
    const u32 off = OFF_XD + (u32)((wave * 32 + g) * 4) * UNIT_B;
    const u32 finfo = (u32)(fv >> 32);
    const u32 ovm = (u32)__builtin_amdgcn_ballot_w64(lane < 4 && (finfo & 0x80) != 0);
    u32x4 pa[4], pb[4], pn[4], po[4];
    u32x2 mk = {0u, 0u};
    if (ovm != 0) {
      mk = __builtin_amdgcn_raw_buffer_load_b64(rsme, fl + 32 + (lane & 3) * 8, 0, AUX_SYS);
#pragma unroll
      for (int u = 0; u < 4; ++u) {
        po[u] = u32x4{0u, 0u, 0u, 0u};
        if ((ovm >> u) & 1) po[u] = __builtin_amdgcn_raw_buffer_load_b128(rsme, off + u * UNIT_B + 3072 + lane * 16, 0, AUX_SYS);
      }
    } else {
#pragma unroll
      for (int u = 0; u < 4; ++u) po[u] = u32x4{0u, 0u, 0u, 0u};
    }
#pragma unroll
    for (int u = 0; u < 4; ++u) {
      pa[u] = __builtin_amdgcn_raw_buffer_load_b128(rsme, off + u * UNIT_B + lane * 16, 0, AUX_SYS);
      pb[u] = __builtin_amdgcn_raw_buffer_load_b128(rsme, off + u * UNIT_B + 1024 + lane * 16, 0, AUX_SYS);
      pn[u] = __builtin_amdgcn_raw_buffer_load_b128(rsme, off + u * UNIT_B + 2048 + lane * 16, 0, AUX_SYS);
    }
    __builtin_amdgcn_sched_barrier(0);
#pragma unroll
    for (int u = 0; u < 4; ++u) {
      const u32 info = __builtin_amdgcn_readlane(finfo, u);
      u32 o[16];
      decode_unit(pa[u], pb[u], pn[u], po[u], info, rdlane64(mk, u), o);
#pragma unroll
      for (int e = 0; e < 4; ++e) {
        const u32x4 xf = {o[4 * e], o[4 * e + 1], o[4 * e + 2], o[4 * e + 3]};
        const int m = 4 * u + e;
        acc0 = __builtin_amdgcn_mfma_f32_32x32x16_bf16(__builtin_bit_cast(bf16x8, xf), __builtin_bit_cast(bf16x8, wf[0][m]), acc0, 0, 0, 0);
        acc1 = __builtin_amdgcn_mfma_f32_32x32x16_bf16(__builtin_bit_cast(bf16x8, xf), __builtin_bit_cast(bf16x8, wf[1][m]), acc1, 0, 0, 0);
      }
    }
  }
  }
  STAMP(5);
  {
    char* red = smem + LDS_RED + wave * 8192;
#pragma unroll
    for (int r = 0; r < 16; ++r) {
      const int row = (r & 3) + 8 * (r >> 2) + 4 * h;
      const int off = (row * 256 + l31 * 8) ^ (((row >> 3) & 1) << 7);
      *(f32x2*)(red + off) = f32x2{acc0[r], acc1[r]};
    }
    const bool bad = !ok || __builtin_amdgcn_readfirstlane(tmv) == call;
    if (lane == 0) ((u32*)(smem + LDS_BAD))[wave] = bad ? 1u : 0u;
  }
  __syncthreads();
  u32 o0, o1, nib, lpre;
  {
    const int cg = lane >> 3, q = (lane >> 1) & 3, hf = lane & 1;
    const int row = 8 * q + wave;
    const int off = (row * 256 + cg * 32 + hf * 16) ^ ((q & 1) << 7);
    f32x4 s = *(const f32x4*)(smem + LDS_RED + off);
#pragma unroll
    for (int w2 = 1; w2 < 8; ++w2) s += *(const f32x4*)(smem + LDS_RED + w2 * 8192 + off);
    const u32x4 bw0 = *(const u32x4*)(smem + LDS_BAD), bw1 = *(const u32x4*)(smem + LDS_BAD + 16);
    const bool bad = (bw0[0] | bw0[1] | bw0[2] | bw0[3] | bw1[0] | bw1[1] | bw1[2] | bw1[3]) != 0;
    float v[4];
#pragma unroll
    for (int e = 0; e < 4; ++e) v[e] = !(s[e] <= 0.f) ? s[e] : 0.f;
    o0 = cvt_pk(v[0], v[1]);
    o1 = cvt_pk(v[2], v[3]);
    if (bad) { o0 = 0x7fc07fc0u; o1 = 0x7fc07fc0u; }
    nib = ((o0 & 0xffffu) ? 1u : 0u) | ((o0 >> 16) ? 2u : 0u) | ((o1 & 0xffffu) ? 4u : 0u) | ((o1 >> 16) ? 8u : 0u);
    const u32 cnt = __builtin_popcount(nib);
    const u32 incl = wave_scan_incl(cnt);
    lpre = incl - cnt;
    const u32 tot = __builtin_amdgcn_readlane(incl, 63);
    const u32 pm = pkmax(o0 & 0x7fff7fffu, o1 & 0x7fff7fffu);
    const u32 mx = wave_max(max(pm & 0xffffu, pm >> 16));
    if (lane == 0) ((u32*)(smem + LDS_INFO))[wave] = tot | (mx << 16);
  }
  __syncthreads();
  u32 count, mx16;
  {
    const u32x4 i0 = *(const u32x4*)(smem + LDS_INFO), i1 = *(const u32x4*)(smem + LDS_INFO + 16);
    u32 pre = 0, tot = 0, mx = 0;
#pragma unroll
    for (int w2 = 0; w2 < 8; ++w2) {
      const u32 iv = w2 < 4 ? i0[w2 & 3] : i1[w2 & 3];
      if (w2 == wave) pre = tot;
      tot += iv & 0xffffu;
      mx = max(mx, iv >> 16);
    }
    count = __builtin_amdgcn_readfirstlane(tot);
    mx16 = __builtin_amdgcn_readfirstlane(mx);
    uint16_t* str = (uint16_t*)(smem + LDS_STR);
    u32 p = pre + lpre;
    if (nib & 1) str[p] = (uint16_t)o0;
    p += nib & 1;
    if (nib & 2) str[p] = (uint16_t)(o0 >> 16);
    p += (nib >> 1) & 1;
    if (nib & 4) str[p] = (uint16_t)o1;
    p += (nib >> 2) & 1;
    if (nib & 8) str[p] = (uint16_t)(o1 >> 16);
    const u32 nnext = (u32)__builtin_amdgcn_update_dpp(0, (int)nib, 0x101  , 0xf, 0xf, false);
    if ((lane & 1) == 0) *(uint8_t*)(smem + LDS_MSK + wave * 32 + (lane >> 1)) = (uint8_t)(nib | (nnext << 4));
    if (wave < 2) str[count + wave * 64 + lane] = (uint16_t)mx16;
  }
  __syncthreads();
  STAMP(6);
  {
    const u32 nv = ((count + 127) >> 7) << 2;
    const u32 lo = lo_from_max(mx16);
    const bool on = (u32)lane < nv;
    const char* sa = smem + LDS_STR + lane * 32;
    const char* sb = sa + nv * 32;
    u32 r[16];
    {
      const u32x4 t0 = *(const u32x4*)(sa), t1 = *(const u32x4*)(sa + 16);
      const u32x4 t2 = *(const u32x4*)(on ? sb : sa), t3 = *(const u32x4*)((on ? sb : sa) + 16);
#pragma unroll
      for (int q = 0; q < 4; ++q) { r[q] = t0[q]; r[4 + q] = t1[q]; r[8 + q] = t2[q]; r[12 + q] = t3[q]; }
    }
    Packed pk;
    u32 ha[4], hb[4];
#pragma unroll
    for (int j = 0; j < 4; ++j) {
      pk.la[j] = perm(r[2 * j + 1], r[2 * j], 0x06040200u);
      ha[j] = perm(r[2 * j + 1], r[2 * j], 0x07050301u);
      pk.lb[j] = perm(r[8 + 2 * j + 1], r[8 + 2 * j], 0x06040200u);
      hb[j] = perm(r[8 + 2 * j + 1], r[8 + 2 * j], 0x07050301u);
    }
    pack_hi(ha, hb, lo, on, pk);
    if (wave == rank) {
      char* ua = smem + LDS_RED + wave * UAREA;
      if (on) {
        *(u32x4*)(ua + lane * 16) = pk.la;
        *(u32x4*)(ua + (nv + lane) * 16) = pk.lb;
        *(u32x4*)(ua + 2176 + lane * 16) = u32x4{ha[0], ha[1], ha[2], ha[3]};
        *(u32x4*)(ua + 2176 + (nv + lane) * 16) = u32x4{hb[0], hb[1], hb[2], hb[3]};
      }
    } else {
      __amdgpu_buffer_rsrc_t rs = mk_rsrc(P.ws[wave], WS_NEED);
      const u32 slot = OFF_OD + (u32)(rank * 256 + g * 8 + c) * OT_B;
      timer_wait(tstart, to0 + g * dO);
      if (lane < 16) {
        const u32x4 mk = *(const u32x4*)(smem + LDS_MSK + lane * 16);
        __builtin_amdgcn_raw_buffer_store_b128(mk, rs, slot + lane * 16, 0, AUX_SYS);
      }
      if (on) {
        __builtin_amdgcn_raw_buffer_store_b128(pk.la, rs, slot + 256 + lane * 16, 0, AUX_SYS);
        __builtin_amdgcn_raw_buffer_store_b128(pk.lb, rs, slot + 256 + (nv + lane) * 16, 0, AUX_SYS);
        __builtin_amdgcn_raw_buffer_store_b128(pk.n, rs, slot + 256 + (2 * nv + lane) * 16, 0, AUX_SYS);
      }
      if (pk.bad != 0) {
        if ((pk.bad >> lane) & 1) __builtin_amdgcn_raw_buffer_store_b128(pk.ovf, rs, slot + OT_OVF + lane * 16, 0, AUX_SYS);
        if (lane == 0) __builtin_amdgcn_raw_buffer_store_b64(u32x2{(u32)pk.bad, (u32)(pk.bad >> 32)}, rs, slot + OT_BM, 0, AUX_SYS);
      }
      asm volatile("s_waitcnt vmcnt(0)" ::: "memory");
      if (lane == 0) {
        const u32x2 f = {call, count | (lo << 12) | (pk.bad != 0 ? 1u << 19 : 0u)};
        __builtin_amdgcn_raw_buffer_store_b64(f, rs, OFF_OF + (rank * 256 + g * 8 + c) * 128, 0, AUX_SYS);
      }
    }
  }
  STAMP(7);
  {
    char* ua = smem + LDS_RED + wave * UAREA;
    u32 m;
    bool bad = false;
    if (wave == rank) {
      m = *(const u32*)(smem + LDS_MSK + lane * 4);
    } else {
      u64 fv;
      const bool ok2 = wait_flags(myws, OFF_OF + (wave * 256 + g * 8 + c) * 128, lane == 0, call, lane, fv, tstart, to0 + g * dO + lo_);
      STAMP(8);
      const u32 tm2 = read_tmo(myws);
      const u32 finfo = __builtin_amdgcn_readlane((u32)(fv >> 32), 0);
      const u32 cnt2 = ok2 ? (finfo & 0xfffu) : 0u;
      const u32 lo4 = ((finfo >> 12) & 0x7f) * 0x01010101u;
      const u32 nv = ((cnt2 + 127) >> 7) << 2;
      const u32 slot = OFF_OD + (u32)(wave * 256 + g * 8 + c) * OT_B;
      m = __builtin_amdgcn_raw_buffer_load_b32(rsme, slot + lane * 4, 0, AUX_SYS);
      u32x4 ov = {0u, 0u, 0u, 0u};
      u32x2 mk = {0u, 0u};
      if (finfo & (1u << 19)) {
        ov = __builtin_amdgcn_raw_buffer_load_b128(rsme, slot + OT_OVF + lane * 16, 0, AUX_SYS);
        mk = __builtin_amdgcn_raw_buffer_load_b64(rsme, slot + OT_BM, 0, AUX_SYS);
      }
      if ((u32)lane < nv) {
        const u32x4 la = __builtin_amdgcn_raw_buffer_load_b128(rsme, slot + 256 + lane * 16, 0, AUX_SYS);
        const u32x4 lb = __builtin_amdgcn_raw_buffer_load_b128(rsme, slot + 256 + (nv + lane) * 16, 0, AUX_SYS);
        const u32x4 n = __builtin_amdgcn_raw_buffer_load_b128(rsme, slot + 256 + (2 * nv + lane) * 16, 0, AUX_SYS);
        u32 ha[4], hb[4];
        unpack_hi(n, lo4, ha, hb);
        if (finfo & (1u << 19)) {
          const bool bd = ((lane < 32 ? mk[0] >> lane : mk[1] >> (lane - 32)) & 1) != 0;
#pragma unroll
          for (int j = 0; j < 4; ++j) {
            ha[j] = bd ? n[j] : ha[j];
            hb[j] = bd ? ov[j] : hb[j];
          }
        }
        *(u32x4*)(ua + lane * 16) = la;
        *(u32x4*)(ua + (nv + lane) * 16) = lb;
        *(u32x4*)(ua + 2176 + lane * 16) = u32x4{ha[0], ha[1], ha[2], ha[3]};
        *(u32x4*)(ua + 2176 + (nv + lane) * 16) = u32x4{hb[0], hb[1], hb[2], hb[3]};
      }
      bad = !ok2 || __builtin_amdgcn_readfirstlane(tm2) == call;
    }
    const u32 pc = __builtin_popcount(m);
    const u32 pos0 = wave_scan_incl(pc) - pc;
    const int si = wave >> 2, sj = wave & 3;
    const u32 obase = ((u32)(1024 * si + 32 * g + (lane >> 3)) * NN + 512 * sj + 64 * c + 8 * (lane & 7)) * 2;
    __amdgpu_buffer_rsrc_t rso = mk_rsrc(P.out, MM * NN * 2);
    asm volatile("" ::: "memory");
    __builtin_amdgcn_wave_barrier();
#pragma unroll
    for (int q = 0; q < 4; ++q) {
      const u32 mq = (m >> (8 * q)) & 0xff;
      u32 pos = pos0 + __builtin_popcount(q == 0 ? 0u : (m & ((1u << (8 * q)) - 1u)));
      pos = min(pos, 2048u);
      const u32x2 sel = *(const u32x2*)(smem + LDS_LUT + mq * 8);
      const u32* lw = (const u32*)(ua + (pos & ~3u));
      const u32* hw = (const u32*)(ua + 2176 + (pos & ~3u));
      const u32 l0 = lw[0], l1 = lw[1], l2 = lw[2];
      const u32 h0 = hw[0], h1 = hw[1], h2 = hw[2];
      const u32 la0 = __builtin_amdgcn_alignbyte(l1, l0, pos), la1 = __builtin_amdgcn_alignbyte(l2, l1, pos);
      const u32 ha0 = __builtin_amdgcn_alignbyte(h1, h0, pos), ha1 = __builtin_amdgcn_alignbyte(h2, h1, pos);
      const u32 eL0 = perm(la1, la0, sel[0]), eL1 = perm(la1, la0, sel[1]);
      const u32 eH0 = perm(ha1, ha0, sel[0]), eH1 = perm(ha1, ha0, sel[1]);
      u32x4 o = {perm(eH0, eL0, 0x05010400u), perm(eH0, eL0, 0x07030602u), perm(eH1, eL1, 0x05010400u), perm(eH1, eL1, 0x07030602u)};
      if (bad) o = u32x4{0x7fc07fc0u, 0x7fc07fc0u, 0x7fc07fc0u, 0x7fc07fc0u};
      __builtin_amdgcn_raw_buffer_store_b128(o, rso, obase + q * 8 * NN * 2, 0, OUT_AUX);
    }
  }
  if (tid == 0) __hip_atomic_store((gu32*)(myws + OFF_CNT + b * 128), call, __ATOMIC_RELAXED, __HIP_MEMORY_SCOPE_AGENT);
  STAMP(9);
}

}

extern "C" void kernel_launch(int rank, int nranks, ncclComm_t comm, hipStream_t stream,
                              void* const* d_in, const int* in_sizes, int n_in,
                              void* d_out, int out_size, const int* in_dtypes, int out_dtype,
                              void* const* d_out_all, void* const* d_ws_all, size_t ws_bytes) {
  if (nranks != NR || ws_bytes < WS_NEED || (unsigned)rank >= (unsigned)NR) {
    fprintf(stderr, "unexpected configuration\n");
    return;
  }
  static bool init[NR] = {};
  if (!init[rank]) {
    HIP_CHECK(hipFuncSetAttribute((const void*)gemm_ar_kernel, hipFuncAttributeMaxDynamicSharedMemorySize, LDS_BYTES));
    init[rank] = true;
  }
  Params p;
  p.x = (const uint16_t*)d_in[0];
  p.w = (const uint16_t*)d_in[1];
  for (int r = 0; r < NR; ++r) p.ws[r] = (char*)d_ws_all[r];
  p.out = (char*)d_out;
  p.rank = rank;
  p.pad = 0;
  hipLaunchKernelGGL(gemm_ar_kernel, dim3(NWG), dim3(NT), LDS_BYTES, stream, p);
}
```

```cpp
#include <hip/hip_runtime.h>
#include <rccl/rccl.h>
#include <cstdio>
#include <cstdint>
#include <cstdlib>

#define HIP_CHECK(expr) do { hipError_t _e = (expr); if (_e != hipSuccess) { \
    fprintf(stderr, "HIP error %s at %s:%d\n", hipGetErrorString(_e), __FILE__, __LINE__); \
    return; } } while (0)

namespace {

typedef unsigned u32;
typedef unsigned long long u64;
typedef u32 u32x4 __attribute__((ext_vector_type(4)));
typedef u32 u32x2 __attribute__((ext_vector_type(2)));
typedef float f32x16 __attribute__((ext_vector_type(16)));
typedef float f32x4 __attribute__((ext_vector_type(4)));
typedef float f32x2 __attribute__((ext_vector_type(2)));
typedef __bf16 bf16x8 __attribute__((ext_vector_type(8)));
typedef __bf16 bf16x2 __attribute__((ext_vector_type(2)));
typedef unsigned short u16x2 __attribute__((ext_vector_type(2)));
typedef __attribute__((address_space(1))) u64 gu64;
typedef __attribute__((address_space(1))) u32 gu32;

constexpr int NR = 8;
constexpr int MM = 2048, NN = 2048, KL = 256;
constexpr int NWG = 256, NT = 512;

constexpr u32 OFF_CNT = 0;
constexpr u32 OFF_TMO = OFF_CNT + NWG * 128;
constexpr u32 OFF_WF = 65536;
constexpr u32 OFF_XF = OFF_WF + 64 * 128;
constexpr u32 OFF_OF = OFF_XF + 256 * 128;
constexpr u32 OFF_DBG = OFF_OF + 2048 * 128;
constexpr u32 UNIT_B = 4096;
constexpr u32 OFF_WD = 1u << 20;
constexpr u32 OFF_XD = OFF_WD + 8 * 8 * 8 * UNIT_B;
constexpr u32 OT_B = 5248;
constexpr u32 OT_OVF = 3072, OT_BM = 5120;
constexpr u32 OFF_OD = OFF_XD + 8 * 32 * 4 * UNIT_B;
constexpr u32 WS_NEED = OFF_OD + 2048 * OT_B;
static_assert(OFF_DBG + 256 * 8 * 16 * 8 <= OFF_WD, "layout");

constexpr u64 DEADLINE = 20000000ull;
constexpr int AUX_SYS = 17;

#define TW0 60
#define DW 45
#define TX0 420
#define DX 25
#define TO0 1210
#define DO 26
#define LW 120
#define LX 120
#define LO 120
#define PSLEEP 4
#define OUT_AUX 16
#define MAPB 1
#define GWFIRST 12

constexpr int LDS_RED = 0;
constexpr int UAREA = 4352;
constexpr int LDS_STR = 65536;
constexpr int LDS_PK = LDS_STR + 4352;
constexpr int LDS_MSK = LDS_PK;
constexpr int LDS_OVF = LDS_PK + 3072;
constexpr int LDS_INFO = LDS_OVF + 2048;
constexpr int LDS_BAD = LDS_INFO + 32;
constexpr int LDS_PBM = LDS_BAD + 32;
constexpr int LDS_LUT = LDS_PBM + 16;
constexpr int LDS_BYTES = LDS_LUT + 2048;
static_assert(8 * UAREA <= 65536, "uarea");

struct Params {
  const uint16_t* x;
  const uint16_t* w;
  char* myws;
  char* out;
  int rank;
  int pad;
  char* ws[NR];
};

__device__ __forceinline__ __amdgpu_buffer_rsrc_t mk_rsrc(const void* p, u32 bytes) {
  return __builtin_amdgcn_make_buffer_rsrc((void*)p, 0, bytes, 0x00020000);
}
__device__ __forceinline__ u32 perm(u32 hi, u32 lo, u32 sel) { return __builtin_amdgcn_perm(hi, lo, sel); }
__device__ __forceinline__ u32 rotr4(u32 v) { return __builtin_amdgcn_alignbit(v, v, 4); }
__device__ __forceinline__ u32 rotl4(u32 v) { return __builtin_amdgcn_alignbit(v, v, 28); }
__device__ __forceinline__ u32 pkmax(u32 a, u32 b) {
  return __builtin_bit_cast(u32, __builtin_elementwise_max(__builtin_bit_cast(u16x2, a), __builtin_bit_cast(u16x2, b)));
}
template <int CTRL, int RM>
__device__ __forceinline__ u32 dpp0(u32 v) { return (u32)__builtin_amdgcn_update_dpp(0, (int)v, CTRL, RM, 0xf, false); }
__device__ __forceinline__ u32 wave_max(u32 v) {
  v = max(v, dpp0<0x111, 0xf>(v));
  v = max(v, dpp0<0x112, 0xf>(v));
  v = max(v, dpp0<0x114, 0xf>(v));
  v = max(v, dpp0<0x118, 0xf>(v));
  v = max(v, dpp0<0x142, 0xa>(v));
  v = max(v, dpp0<0x143, 0xc>(v));
  return (u32)__builtin_amdgcn_readlane((int)v, 63);
}
__device__ __forceinline__ u32 wave_scan_incl(u32 v) {
  v += dpp0<0x111, 0xf>(v);
  v += dpp0<0x112, 0xf>(v);
  v += dpp0<0x114, 0xf>(v);
  v += dpp0<0x118, 0xf>(v);
  v += dpp0<0x142, 0xa>(v);
  v += dpp0<0x143, 0xc>(v);
  return v;
}

#define STAMP(i) do {} while (0)
#define STAMP2(i) do {} while (0)

__device__ __forceinline__ bool wait_flags(char* myws, u32 off, bool act, u32 call, int lane, u64& val, u64 tstart, u32 notbefore) {
  gu64* addr = (gu64*)(myws + off);
  while (__builtin_amdgcn_s_memrealtime() - tstart < (u64)notbefore) __builtin_amdgcn_s_sleep(2);
  const u64 t0 = __builtin_amdgcn_s_memrealtime();
  bool ok = true;
  for (;;) {
    u64 v = (u64)call;
    if (act) v = __hip_atomic_load(addr, __ATOMIC_RELAXED, __HIP_MEMORY_SCOPE_SYSTEM);
    val = v;
    const u64 pend = __builtin_amdgcn_ballot_w64(act && (u32)v < call);
    if (pend == 0) break;
    const u64 el = __builtin_amdgcn_s_memrealtime() - t0;
    if (el > 50000) {
      const u32 tm = (u32)__hip_atomic_load((gu64*)(myws + OFF_TMO), __ATOMIC_RELAXED, __HIP_MEMORY_SCOPE_AGENT);
      if (__builtin_amdgcn_readfirstlane(tm) == call || el > DEADLINE) {
        if (lane == 0) __hip_atomic_store((gu64*)(myws + OFF_TMO), (u64)call, __ATOMIC_RELAXED, __HIP_MEMORY_SCOPE_AGENT);
        ok = false;
        break;
      }
    }
    __builtin_amdgcn_s_sleep(PSLEEP);
  }
  asm volatile("" ::: "memory");
  return ok;
}
__device__ __forceinline__ u32 read_tmo(char* myws) {
  return (u32)__hip_atomic_load((gu64*)(myws + OFF_TMO), __ATOMIC_RELAXED, __HIP_MEMORY_SCOPE_AGENT);
}

__device__ __forceinline__ u32 cvt_pk(float a, float b_) {
  f32x2 f = {a, b_};
  return __builtin_bit_cast(u32, __builtin_convertvector(f, bf16x2));
}

struct Packed { u32x4 la, lb, n, ovf; u64 bad; };

__device__ __forceinline__ void pack_hi(const u32 (&ha)[4], const u32 (&hb)[4], u32 lo, bool lane_on, Packed& pk) {
  const u32 lo4 = lo * 0x01010101u;
  u32 acc = 0x80808080u;
#pragma unroll
  for (int j = 0; j < 4; ++j) {
    const u32 da = (ha[j] | 0x80808080u) - lo4;
    const u32 db = (hb[j] | 0x80808080u) - lo4;
    acc &= da & db;
    const u32 xa = (ha[j] & 0x80808080u) | (da & 0x7f7f7f7fu);
    const u32 xb = (hb[j] & 0x80808080u) | (db & 0x7f7f7f7fu);
    pk.n[j] = xa | rotr4(xb);
  }
  const bool bad = lane_on && acc != 0x80808080u;
  pk.bad = __builtin_amdgcn_ballot_w64(bad);
  if (pk.bad != 0) {
#pragma unroll
    for (int j = 0; j < 4; ++j) {
      pk.n[j] = bad ? ha[j] : pk.n[j];
      pk.ovf[j] = hb[j];
    }
  }
}
__device__ __forceinline__ void unpack_hi(const u32x4& n, u32 lo4, u32 (&ha)[4], u32 (&hb)[4]) {
#pragma unroll
  for (int j = 0; j < 4; ++j) {
    ha[j] = (n[j] & 0x87878787u) + lo4;
    hb[j] = (rotl4(n[j]) & 0x87878787u) + lo4;
  }
}
typedef u32 u32x3 __attribute__((ext_vector_type(3)));
__device__ __forceinline__ u64 pack3(const u32 (&ha)[4], const u32 (&hb)[4], u32 lo, bool lane_on, u32 (&n)[3]) {
  const u32 lo4 = lo * 0x01010101u;
  u32 acc = 0x80808080u, sgn = 0, x[8];
#pragma unroll
  for (int j = 0; j < 4; ++j) {
    const u32 da = (ha[j] | 0x80808080u) - lo4;
    const u32 db = (hb[j] | 0x80808080u) - lo4;
    acc &= da & db;
    sgn |= ha[j] | hb[j];
    x[j] = da & 0x07070707u;
    x[4 + j] = db & 0x07070707u;
  }
  n[0] = x[0] | (x[1] << 3) | ((x[6] & 0x03030303u) << 6);
  n[1] = x[2] | (x[3] << 3) | ((x[7] & 0x03030303u) << 6);
  n[2] = x[4] | (x[5] << 3) | ((x[6] & 0x04040404u) << 4) | ((x[7] & 0x04040404u) << 5);
  const bool bad = lane_on && (acc != 0x80808080u || (sgn & 0x80808080u) != 0);
  return __builtin_amdgcn_ballot_w64(bad);
}
__device__ __forceinline__ void unpack3(const u32x3& n, u32 lo4, u32 (&ha)[4], u32 (&hb)[4]) {
  ha[0] = (n[0] & 0x07070707u) + lo4;
  ha[1] = ((n[0] >> 3) & 0x07070707u) + lo4;
  ha[2] = (n[1] & 0x07070707u) + lo4;
  ha[3] = ((n[1] >> 3) & 0x07070707u) + lo4;
  hb[0] = (n[2] & 0x07070707u) + lo4;
  hb[1] = ((n[2] >> 3) & 0x07070707u) + lo4;
  hb[2] = (((n[0] >> 6) & 0x03030303u) | ((n[2] >> 4) & 0x04040404u)) + lo4;
  hb[3] = (((n[1] >> 6) & 0x03030303u) | ((n[2] >> 5) & 0x04040404u)) + lo4;
}
__device__ __forceinline__ u32 lo_from_max(u32 m16) {
  const u32 e7 = m16 >> 8;
  return e7 > 7 ? e7 - 7 : 0;
}

__device__ __forceinline__ void timer_wait(u64 tstart, u32 dly) {
  while (__builtin_amdgcn_s_memrealtime() - tstart < (u64)dly) __builtin_amdgcn_s_sleep(1);
}

__device__ __forceinline__ u32 call_no(u32 cntv) {
  asm volatile("" : "+v"(cntv));
  return __builtin_amdgcn_readfirstlane(cntv) + 1;
}

template <int ND>
__device__ __forceinline__ void send_unit(const Params& P, const Packed& pk, u32 lo, int dst0, int dstep, u32 doff,
                                          u32 foff, u32 moff, u32 callv, int lane, u64 tstart, u32 dly) {
  STAMP2(10 + (ND == 4 ? 2 : 0));
  timer_wait(tstart, dly);
  STAMP2(14 + (ND == 4 ? 1 : 0));
#pragma unroll
  for (int d = 0; d < ND; ++d) {
    if (dst0 + d * dstep == P.rank) continue;
    __amdgpu_buffer_rsrc_t rs = mk_rsrc(P.ws[dst0 + d * dstep], WS_NEED);
    __builtin_amdgcn_raw_buffer_store_b128(pk.la, rs, doff + lane * 16, 0, AUX_SYS);
    __builtin_amdgcn_raw_buffer_store_b128(pk.lb, rs, doff + 1024 + lane * 16, 0, AUX_SYS);
    __builtin_amdgcn_raw_buffer_store_b128(pk.n, rs, doff + 2048 + lane * 16, 0, AUX_SYS);
  }
  if (pk.bad != 0) {
#pragma unroll
    for (int d = 0; d < ND; ++d) {
      if (dst0 + d * dstep == P.rank) continue;
      __amdgpu_buffer_rsrc_t rs = mk_rsrc(P.ws[dst0 + d * dstep], WS_NEED);
      if ((pk.bad >> lane) & 1) __builtin_amdgcn_raw_buffer_store_b128(pk.ovf, rs, doff + 3072 + lane * 16, 0, AUX_SYS);
      if (lane == 0) __builtin_amdgcn_raw_buffer_store_b64(u32x2{(u32)pk.bad, (u32)(pk.bad >> 32)}, rs, moff, 0, AUX_SYS);
    }
  }
  asm volatile("s_waitcnt vmcnt(0)" ::: "memory");
  STAMP2(11 + (ND == 4 ? 2 : 0));
  const u32x2 f = {call_no(callv), lo | (pk.bad != 0 ? 0x80u : 0u)};
  if (lane == 0) {
#pragma unroll
    for (int d = 0; d < ND; ++d)
      if (dst0 + d * dstep != P.rank)
        __builtin_amdgcn_raw_buffer_store_b64(f, mk_rsrc(P.ws[dst0 + d * dstep], WS_NEED), foff, 0, AUX_SYS);
  }
}

__device__ __forceinline__ void send_x(const Params& P, const uint16_t* px, int g, int c, u32 call, int lane, u64 tstart, u32 dly) {
  const int rank = P.rank, h = lane >> 5, l31 = lane & 31;
  const int ih = c >> 2, u = c & 3;
  __amdgpu_buffer_rsrc_t rx = mk_rsrc(px, MM * KL * 2);
  const u32 voff = (u32)l31 * (KL * 2) + 256 * h;
  const u32 soff = (u32)(1024 * ih + 32 * g) * (KL * 2) + 64 * u;
  u32 r[16];
#pragma unroll
  for (int p = 0; p < 4; ++p) {
    const u32x4 t = __builtin_amdgcn_raw_buffer_load_b128(rx, voff + p * 16, soff, 0);
#pragma unroll
    for (int q = 0; q < 4; ++q) r[4 * p + q] = t[q];
  }
  u32 m = 0;
#pragma unroll
  for (int t = 0; t < 16; ++t) m = pkmax(m, r[t] & 0x7fff7fffu);
  const u32 lo = lo_from_max(wave_max(max(m & 0xffffu, m >> 16)));
  Packed pk;
  u32 ha[4], hb[4];
#pragma unroll
  for (int j = 0; j < 4; ++j) {
    pk.la[j] = perm(r[2 * j + 1], r[2 * j], 0x06040200u);
    ha[j] = perm(r[2 * j + 1], r[2 * j], 0x07050301u);
    pk.lb[j] = perm(r[8 + 2 * j + 1], r[8 + 2 * j], 0x06040200u);
    hb[j] = perm(r[8 + 2 * j + 1], r[8 + 2 * j], 0x07050301u);
  }
  pack_hi(ha, hb, lo, true, pk);
  const u32 fl = OFF_XF + (rank * 32 + g) * 128;
  send_unit<4>(P, pk, lo, 4 * ih, 1, OFF_XD + (u32)((rank * 32 + g) * 4 + u) * UNIT_B, fl + u * 8, fl + 32 + u * 8, call, lane, tstart, dly);
}

__device__ __forceinline__ void send_w(const Params& P, const uint16_t* pw, int jd, int cd, int u8, u32 call, int lane, u64 tstart, u32 dly) {
  const int rank = P.rank, h = lane >> 5, l31 = lane & 31;
  const int cgl = 8 * jd + cd;
  __amdgpu_buffer_rsrc_t rw = mk_rsrc(pw, KL * NN * 2);
  const u32 voff = (u32)(128 * h) * (NN * 2) + 4 * l31;
  const u32 soff = (u32)(16 * u8) * (NN * 2) + 128 * cgl;
  u32 r[16];
#pragma unroll
  for (int k = 0; k < 16; ++k) r[k] = __builtin_amdgcn_raw_buffer_load_b32(rw, voff, soff + k * (NN * 2), 0);
  u32 m = 0;
#pragma unroll
  for (int t = 0; t < 16; ++t) m = pkmax(m, r[t] & 0x7fff7fffu);
  const u32 lo = lo_from_max(wave_max(max(m & 0xffffu, m >> 16)));
  Packed pk;
  u32 ha[4], hb[4];
#pragma unroll
  for (int j = 0; j < 4; ++j) {
    const u32 t0 = perm(r[4 * j + 1], r[4 * j], 0x05010400u), t1 = perm(r[4 * j + 1], r[4 * j], 0x07030602u);
    const u32 t2 = perm(r[4 * j + 3], r[4 * j + 2], 0x05010400u), t3 = perm(r[4 * j + 3], r[4 * j + 2], 0x07030602u);
    pk.la[j] = perm(t2, t0, 0x05040100u);
    ha[j] = perm(t2, t0, 0x07060302u);
    pk.lb[j] = perm(t3, t1, 0x05040100u);
    hb[j] = perm(t3, t1, 0x07060302u);
  }
  pack_hi(ha, hb, lo, true, pk);
  const u32 fl = OFF_WF + (rank * 8 + cd) * 128 + (u8 >> 2) * 64 + (u8 & 3) * 8;
  send_unit<2>(P, pk, lo, jd, 4, OFF_WD + (u32)((rank * 8 + cd) * 8 + u8) * UNIT_B, fl, fl + 32, call, lane, tstart, dly);
}

__device__ __forceinline__ void decode_unit(const u32x4& la, const u32x4& lb, const u32x4& n, const u32x4& ov, u32 info,
                                            u64 badmask, u32 (&o)[16]) {
  const u32 lo4 = (info & 0x7f) * 0x01010101u;
  u32 ha[4], hb[4];
  unpack_hi(n, lo4, ha, hb);
  if (info & 0x80) {
    const bool bad = __builtin_amdgcn_inverse_ballot_w64(badmask);
#pragma unroll
    for (int j = 0; j < 4; ++j) {
      ha[j] = bad ? n[j] : ha[j];
      hb[j] = bad ? ov[j] : hb[j];
    }
  }
#pragma unroll
  for (int j = 0; j < 4; ++j) {
    o[2 * j] = perm(ha[j], la[j], 0x05010400u);
    o[2 * j + 1] = perm(ha[j], la[j], 0x07030602u);
    o[8 + 2 * j] = perm(hb[j], lb[j], 0x05010400u);
    o[8 + 2 * j + 1] = perm(hb[j], lb[j], 0x07030602u);
  }
}
__device__ __forceinline__ u64 rdlane64(const u32x2& v, int l) {
  return (u64)(u32)__builtin_amdgcn_readlane(v[0], l) | ((u64)(u32)__builtin_amdgcn_readlane(v[1], l) << 32);
}

__global__ __launch_bounds__(NT, 2) void gemm_ar_kernel(Params P) {
  extern __shared__ __attribute__((aligned(16))) char smem[];
  const int tid = threadIdx.x;
  const int lane = tid & 63;
  const int wave = __builtin_amdgcn_readfirstlane(tid >> 6);
  const int b = blockIdx.x;
  int rank = P.rank;
  const int g = b >> 3, c = b & 7;
  char* myws = P.myws;
  const int h = lane >> 5, l31 = lane & 31;
  const uint16_t* pw = P.w;
  const uint16_t* px = P.x;
  asm volatile("" : "+s"(pw), "+s"(px), "+s"(myws), "+s"(rank));
  const u64 tstart = __builtin_amdgcn_s_memrealtime();
  STAMP(0);

  const u32 callv = __hip_atomic_load((gu32*)(myws + OFF_CNT + b * 128), __ATOMIC_RELAXED, __HIP_MEMORY_SCOPE_AGENT);
  bool ok = true;

  constexpr u32 tw0 = TW0, dw = DW, tx0 = TX0, dx = DX, to0 = TO0, dO = DO;
  constexpr int gwfirst = GWFIRST;
  constexpr u32 lw = LW, lx = LX, lo_ = LO;
  __amdgpu_buffer_rsrc_t rsme = mk_rsrc(myws, WS_NEED);
  f32x16 acc0 = {}, acc1 = {};
  u32 tmv = 0;
  if (wave == rank) {
    send_w(P, pw, g & 3, g >> 2, c, callv, lane, tstart, tw0 + (g >> 2) * dw);
    if (g < gwfirst) send_x(P, px, g, c, callv, lane, tstart, tx0 + g * dx);
    STAMP(2);
    u32x4 wf[2][16];
    {
      __amdgpu_buffer_rsrc_t rw = mk_rsrc(pw, KL * NN * 2);
      const u32 voff = (u32)(128 * h) * (NN * 2) + 4 * l31;
      const u32 soff = (512 * (rank & 3) + 64 * c) * 2;
#pragma unroll
      for (int m2 = 0; m2 < 2; ++m2) {
        u32 r[64];
#pragma unroll
        for (int k = 0; k < 64; ++k) r[k] = __builtin_amdgcn_raw_buffer_load_b32(rw, voff, soff + (64 * m2 + k) * (NN * 2), 0);
        __builtin_amdgcn_sched_barrier(0);
#pragma unroll
        for (int e = 0; e < 8; ++e)
#pragma unroll
          for (int q = 0; q < 4; ++q) {
            wf[0][8 * m2 + e][q] = perm(r[8 * e + 2 * q + 1], r[8 * e + 2 * q], 0x05040100u);
            wf[1][8 * m2 + e][q] = perm(r[8 * e + 2 * q + 1], r[8 * e + 2 * q], 0x07060302u);
          }
      }
    }
    STAMP(3);
    {
      __amdgpu_buffer_rsrc_t rx = mk_rsrc(px, MM * KL * 2);
      const u32 voff = (u32)l31 * (KL * 2) + 256 * h;
      const u32 soff = (u32)(1024 * (rank >> 2) + 32 * g) * (KL * 2);
      u32x4 xf[16];
#pragma unroll
      for (int m = 0; m < 16; ++m) xf[m] = __builtin_amdgcn_raw_buffer_load_b128(rx, voff + m * 16, soff, 0);
      __builtin_amdgcn_sched_barrier(0);
#pragma unroll
      for (int m = 0; m < 16; ++m) {
        acc0 = __builtin_amdgcn_mfma_f32_32x32x16_bf16(__builtin_bit_cast(bf16x8, xf[m]), __builtin_bit_cast(bf16x8, wf[0][m]), acc0, 0, 0, 0);
        acc1 = __builtin_amdgcn_mfma_f32_32x32x16_bf16(__builtin_bit_cast(bf16x8, xf[m]), __builtin_bit_cast(bf16x8, wf[1][m]), acc1, 0, 0, 0);
      }
    }
    STAMP(4);
    if (g >= gwfirst) send_x(P, px, g, c, callv, lane, tstart, tx0 + g * dx);
    tmv = read_tmo(myws);
  } else {
    const u32 lw4 = (wave - rank - 1) & 7;
    if (lw4 < 4) {
      const u32 m = lw4 * 64 + lane;
      u32 sel[2] = {0u, 0u};
      u32 cnt = 0;
#pragma unroll
      for (int j = 0; j < 8; ++j) {
        const u32 bit = (m >> j) & 1;
        const u32 s = bit ? cnt : 0x0cu;
        sel[j >> 2] |= s << (8 * (j & 3));
        cnt += bit;
      }
      *(u32x2*)(smem + LDS_LUT + m * 8) = u32x2{sel[0], sel[1]};
    }
    STAMP(1);
    const u32 call = call_no(callv);
    u32x4 wf[2][16];
  {
    u64 fv;
    const u32 fl = OFF_WF + (wave * 8 + c) * 128;
    ok = wait_flags(myws, fl + (lane >> 2) * 64 + (lane & 3) * 8, lane < 8, call, lane, fv, tstart, max(tw0 + c * dw, 100u) + lw) && ok;
    STAMP(2);
    const u32 off = OFF_WD + (u32)((wave * 8 + c) * 8) * UNIT_B;
    const u32 finfo = (u32)(fv >> 32);
    const u32 ovm = (u32)__builtin_amdgcn_ballot_w64(lane < 8 && (finfo & 0x80) != 0);
    u32x4 pa[8], pb[8], pn[8], po[8];
    u32x2 mk = {0u, 0u};
    if (ovm != 0) {
      mk = __builtin_amdgcn_raw_buffer_load_b64(rsme, fl + ((lane >> 2) & 1) * 64 + 32 + (lane & 3) * 8, 0, AUX_SYS);
#pragma unroll
      for (int u8 = 0; u8 < 8; ++u8) {
        po[u8] = u32x4{0u, 0u, 0u, 0u};
        if ((ovm >> u8) & 1) po[u8] = __builtin_amdgcn_raw_buffer_load_b128(rsme, off + u8 * UNIT_B + 3072 + lane * 16, 0, AUX_SYS);
      }
    } else {
#pragma unroll
      for (int u8 = 0; u8 < 8; ++u8) po[u8] = u32x4{0u, 0u, 0u, 0u};
    }
#pragma unroll
    for (int u8 = 0; u8 < 8; ++u8) {
      pa[u8] = __builtin_amdgcn_raw_buffer_load_b128(rsme, off + u8 * UNIT_B + lane * 16, 0, AUX_SYS);
      pb[u8] = __builtin_amdgcn_raw_buffer_load_b128(rsme, off + u8 * UNIT_B + 1024 + lane * 16, 0, AUX_SYS);
      pn[u8] = __builtin_amdgcn_raw_buffer_load_b128(rsme, off + u8 * UNIT_B + 2048 + lane * 16, 0, AUX_SYS);
    }
    __builtin_amdgcn_sched_barrier(0);
#pragma unroll
    for (int u8 = 0; u8 < 8; ++u8) {
      const u32 info = __builtin_amdgcn_readlane(finfo, u8);
      u32 o[16];
      decode_unit(pa[u8], pb[u8], pn[u8], po[u8], info, rdlane64(mk, u8), o);
#pragma unroll
      for (int e = 0; e < 2; ++e)
#pragma unroll
        for (int q = 0; q < 4; ++q) {
          wf[0][2 * u8 + e][q] = o[4 * e + q];
          wf[1][2 * u8 + e][q] = o[8 + 4 * e + q];
        }
    }
  }
  STAMP(3);
  {
    u64 fv;
    const u32 fl = OFF_XF + (wave * 32 + g) * 128;
    ok = wait_flags(myws, fl + lane * 8, lane < 4, call, lane, fv, tstart, tx0 + g * dx + lx) && ok;
    STAMP(4);
    tmv = read_tmo(myws);
    const u32 off = OFF_XD + (u32)((wave * 32 + g) * 4) * UNIT_B;
    const u32 finfo = (u32)(fv >> 32);
    const u32 ovm = (u32)__builtin_amdgcn_ballot_w64(lane < 4 && (finfo & 0x80) != 0);
    u32x4 pa[4], pb[4], pn[4], po[4];
    u32x2 mk = {0u, 0u};
    if (ovm != 0) {
      mk = __builtin_amdgcn_raw_buffer_load_b64(rsme, fl + 32 + (lane & 3) * 8, 0, AUX_SYS);
#pragma unroll
      for (int u = 0; u < 4; ++u) {
        po[u] = u32x4{0u, 0u, 0u, 0u};
        if ((ovm >> u) & 1) po[u] = __builtin_amdgcn_raw_buffer_load_b128(rsme, off + u * UNIT_B + 3072 + lane * 16, 0, AUX_SYS);
      }
    } else {
#pragma unroll
      for (int u = 0; u < 4; ++u) po[u] = u32x4{0u, 0u, 0u, 0u};
    }
#pragma unroll
    for (int u = 0; u < 4; ++u) {
      pa[u] = __builtin_amdgcn_raw_buffer_load_b128(rsme, off + u * UNIT_B + lane * 16, 0, AUX_SYS);
      pb[u] = __builtin_amdgcn_raw_buffer_load_b128(rsme, off + u * UNIT_B + 1024 + lane * 16, 0, AUX_SYS);
      pn[u] = __builtin_amdgcn_raw_buffer_load_b128(rsme, off + u * UNIT_B + 2048 + lane * 16, 0, AUX_SYS);
    }
    __builtin_amdgcn_sched_barrier(0);
#pragma unroll
    for (int u = 0; u < 4; ++u) {
      const u32 info = __builtin_amdgcn_readlane(finfo, u);
      u32 o[16];
      decode_unit(pa[u], pb[u], pn[u], po[u], info, rdlane64(mk, u), o);
#pragma unroll
      for (int e = 0; e < 4; ++e) {
        const u32x4 xf = {o[4 * e], o[4 * e + 1], o[4 * e + 2], o[4 * e + 3]};
        const int m = 4 * u + e;
        acc0 = __builtin_amdgcn_mfma_f32_32x32x16_bf16(__builtin_bit_cast(bf16x8, xf), __builtin_bit_cast(bf16x8, wf[0][m]), acc0, 0, 0, 0);
        acc1 = __builtin_amdgcn_mfma_f32_32x32x16_bf16(__builtin_bit_cast(bf16x8, xf), __builtin_bit_cast(bf16x8, wf[1][m]), acc1, 0, 0, 0);
      }
    }
  }
  }
  STAMP(5);
  const u32 call = call_no(callv);
  {
    char* red = smem + LDS_RED + wave * 8192;
#pragma unroll
    for (int r = 0; r < 16; ++r) {
      const int row = (r & 3) + 8 * (r >> 2) + 4 * h;
      const int off = (row * 256 + l31 * 8) ^ (((row >> 3) & 1) << 7);
      *(f32x2*)(red + off) = f32x2{acc0[r], acc1[r]};
    }
    const bool bad = !ok || __builtin_amdgcn_readfirstlane(tmv) == call;
    if (lane == 0) ((u32*)(smem + LDS_BAD))[wave] = bad ? 1u : 0u;
  }
  __syncthreads();
  u32 o0, o1, nib, lpre;
  {
    const int cg = lane >> 3, q = (lane >> 1) & 3, hf = lane & 1;
    const int row = 8 * q + wave;
    const int off = (row * 256 + cg * 32 + hf * 16) ^ ((q & 1) << 7);
    f32x4 s = *(const f32x4*)(smem + LDS_RED + off);
#pragma unroll
    for (int w2 = 1; w2 < 8; ++w2) s += *(const f32x4*)(smem + LDS_RED + w2 * 8192 + off);
    const u32x4 bw0 = *(const u32x4*)(smem + LDS_BAD), bw1 = *(const u32x4*)(smem + LDS_BAD + 16);
    const bool bad = (bw0[0] | bw0[1] | bw0[2] | bw0[3] | bw1[0] | bw1[1] | bw1[2] | bw1[3]) != 0;
    float v[4];
#pragma unroll
    for (int e = 0; e < 4; ++e) v[e] = !(s[e] <= 0.f) ? s[e] : 0.f;
    o0 = cvt_pk(v[0], v[1]);
    o1 = cvt_pk(v[2], v[3]);
    if (bad) { o0 = 0x7fc07fc0u; o1 = 0x7fc07fc0u; }
    nib = ((o0 & 0xffffu) ? 1u : 0u) | ((o0 >> 16) ? 2u : 0u) | ((o1 & 0xffffu) ? 4u : 0u) | ((o1 >> 16) ? 8u : 0u);
    const u32 cnt = __builtin_popcount(nib);
    const u32 incl = wave_scan_incl(cnt);
    lpre = incl - cnt;
    const u32 tot = __builtin_amdgcn_readlane(incl, 63);
    const u32 pm = pkmax(o0 & 0x7fff7fffu, o1 & 0x7fff7fffu);
    const u32 mx = wave_max(max(pm & 0xffffu, pm >> 16));
    if (lane == 0) ((u32*)(smem + LDS_INFO))[wave] = tot | (mx << 16);
  }
  __syncthreads();
  u32 count, mx16;
  {
    const u32x4 i0 = *(const u32x4*)(smem + LDS_INFO), i1 = *(const u32x4*)(smem + LDS_INFO + 16);
    u32 pre = 0, tot = 0, mx = 0;
#pragma unroll
    for (int w2 = 0; w2 < 8; ++w2) {
      const u32 iv = w2 < 4 ? i0[w2 & 3] : i1[w2 & 3];
      if (w2 == wave) pre = tot;
      tot += iv & 0xffffu;
      mx = max(mx, iv >> 16);
    }
    count = __builtin_amdgcn_readfirstlane(tot);
    mx16 = __builtin_amdgcn_readfirstlane(mx);
    uint16_t* str = (uint16_t*)(smem + LDS_STR);
    u32 p = pre + lpre;
    if (nib & 1) str[p] = (uint16_t)o0;
    p += nib & 1;
    if (nib & 2) str[p] = (uint16_t)(o0 >> 16);
    p += (nib >> 1) & 1;
    if (nib & 4) str[p] = (uint16_t)o1;
    p += (nib >> 2) & 1;
    if (nib & 8) str[p] = (uint16_t)(o1 >> 16);
    const u32 nnext = (u32)__builtin_amdgcn_update_dpp(0, (int)nib, 0x101  , 0xf, 0xf, false);
    if ((lane & 1) == 0) *(uint8_t*)(smem + LDS_MSK + wave * 32 + (lane >> 1)) = (uint8_t)(nib | (nnext << 4));
    if (wave == 0) str[count + lane] = (uint16_t)mx16;
  }
  __syncthreads();
  STAMP(6);
  const u32 nvs = (count + 31) >> 5;
  const u32 los = lo_from_max(mx16);
  if (wave == rank) {
    const u32 nv = nvs;
    const bool on = (u32)lane < nv;
    const char* sa = smem + LDS_STR + lane * 32;
    const char* sb = on ? sa + nv * 32 : sa;
    u32 r[16];
    {
      const u32x4 t0 = *(const u32x4*)(sa), t1 = *(const u32x4*)(sa + 16);
      const u32x4 t2 = *(const u32x4*)(sb), t3 = *(const u32x4*)(sb + 16);
#pragma unroll
      for (int q = 0; q < 4; ++q) { r[q] = t0[q]; r[4 + q] = t1[q]; r[8 + q] = t2[q]; r[12 + q] = t3[q]; }
    }
    u32x4 la, lb;
    u32 ha[4], hb[4], n[3];
#pragma unroll
    for (int j = 0; j < 4; ++j) {
      la[j] = perm(r[2 * j + 1], r[2 * j], 0x06040200u);
      ha[j] = perm(r[2 * j + 1], r[2 * j], 0x07050301u);
      lb[j] = perm(r[8 + 2 * j + 1], r[8 + 2 * j], 0x06040200u);
      hb[j] = perm(r[8 + 2 * j + 1], r[8 + 2 * j], 0x07050301u);
    }
    const u64 bm = pack3(ha, hb, los, on, n);
    char* ua = smem + LDS_RED + wave * UAREA;
    char* pk = smem + LDS_PK + 256;
    if (on) {
      *(u32x4*)(pk + lane * 16) = la;
      *(u32x4*)(pk + (nv + lane) * 16) = lb;
      u32* pn = (u32*)(pk + nv * 32 + lane * 12);
      pn[0] = n[0]; pn[1] = n[1]; pn[2] = n[2];
      *(u32x4*)(ua + lane * 16) = la;
      *(u32x4*)(ua + (nv + lane) * 16) = lb;
      *(u32x4*)(ua + 2176 + lane * 16) = u32x4{ha[0], ha[1], ha[2], ha[3]};
      *(u32x4*)(ua + 2176 + (nv + lane) * 16) = u32x4{hb[0], hb[1], hb[2], hb[3]};
    }
    if (bm != 0) {
      *(u32x4*)(smem + LDS_OVF + lane * 32) = u32x4{ha[0], ha[1], ha[2], ha[3]};
      *(u32x4*)(smem + LDS_OVF + lane * 32 + 16) = u32x4{hb[0], hb[1], hb[2], hb[3]};
    }
    if (lane == 0) *(u32x2*)(smem + LDS_PBM) = u32x2{(u32)bm, (u32)(bm >> 32)};
  }
  __syncthreads();
  if (wave != rank) {
    const u32x2 bmv = *(const u32x2*)(smem + LDS_PBM);
    const u64 bm = ((u64)(u32)__builtin_amdgcn_readfirstlane(bmv[1]) << 32) | (u32)__builtin_amdgcn_readfirstlane(bmv[0]);
    const u32 tb = 256 + 44 * nvs;
    __amdgpu_buffer_rsrc_t rs = mk_rsrc(P.ws[wave], WS_NEED);
    const u32 slot = OFF_OD + (u32)(rank * 256 + g * 8 + c) * OT_B;
    u32x4 v[3];
#pragma unroll
    for (int k = 0; k < 3; ++k) v[k] = *(const u32x4*)(smem + LDS_PK + (k * 64 + lane) * 16);
    timer_wait(tstart, to0 + g * dO);
#pragma unroll
    for (int k = 0; k < 3; ++k)
      if ((u32)(k * 64 + lane) * 16 < tb) __builtin_amdgcn_raw_buffer_store_b128(v[k], rs, slot + (k * 64 + lane) * 16, 0, AUX_SYS);
    if (bm != 0) {
      if ((bm >> lane) & 1) {
        __builtin_amdgcn_raw_buffer_store_b128(*(const u32x4*)(smem + LDS_OVF + lane * 32), rs, slot + OT_OVF + lane * 32, 0, AUX_SYS);
        __builtin_amdgcn_raw_buffer_store_b128(*(const u32x4*)(smem + LDS_OVF + lane * 32 + 16), rs, slot + OT_OVF + lane * 32 + 16, 0, AUX_SYS);
      }
      if (lane == 0) __builtin_amdgcn_raw_buffer_store_b64(bmv, rs, slot + OT_BM, 0, AUX_SYS);
    }
    asm volatile("s_waitcnt vmcnt(0)" ::: "memory");
    if (lane == 0) {
      const u32x2 f = {call, count | (los << 12) | (bm != 0 ? 1u << 19 : 0u)};
      __builtin_amdgcn_raw_buffer_store_b64(f, rs, OFF_OF + (rank * 256 + g * 8 + c) * 128, 0, AUX_SYS);
    }
  }
  STAMP(7);
  {
    char* ua = smem + LDS_RED + wave * UAREA;
    u32 m;
    bool bad = false;
    if (wave == rank) {
      m = *(const u32*)(smem + LDS_MSK + lane * 4);
    } else {
      u64 fv;
      const bool ok2 = wait_flags(myws, OFF_OF + (wave * 256 + g * 8 + c) * 128, lane == 0, call, lane, fv, tstart, to0 + g * dO + lo_);
      STAMP(8);
      const u32 tm2 = read_tmo(myws);
      const u32 finfo = __builtin_amdgcn_readlane((u32)(fv >> 32), 0);
      const u32 cnt2 = ok2 ? (finfo & 0xfffu) : 0u;
      const u32 lo4 = ((finfo >> 12) & 0x7f) * 0x01010101u;
      const u32 nv = (cnt2 + 31) >> 5;
      const u32 slot = OFF_OD + (u32)(wave * 256 + g * 8 + c) * OT_B;
      m = __builtin_amdgcn_raw_buffer_load_b32(rsme, slot + lane * 4, 0, AUX_SYS);
      u32x4 ova = {0u, 0u, 0u, 0u}, ovb = {0u, 0u, 0u, 0u};
      u32x2 mk = {0u, 0u};
      if (finfo & (1u << 19)) {
        ova = __builtin_amdgcn_raw_buffer_load_b128(rsme, slot + OT_OVF + lane * 32, 0, AUX_SYS);
        ovb = __builtin_amdgcn_raw_buffer_load_b128(rsme, slot + OT_OVF + lane * 32 + 16, 0, AUX_SYS);
        mk = __builtin_amdgcn_raw_buffer_load_b64(rsme, slot + OT_BM, 0, AUX_SYS);
      }
      if ((u32)lane < nv) {
        const u32x4 la = __builtin_amdgcn_raw_buffer_load_b128(rsme, slot + 256 + lane * 16, 0, AUX_SYS);
        const u32x4 lb = __builtin_amdgcn_raw_buffer_load_b128(rsme, slot + 256 + (nv + lane) * 16, 0, AUX_SYS);
        const u32x3 n = __builtin_amdgcn_raw_buffer_load_b96(rsme, slot + 256 + nv * 32 + lane * 12, 0, AUX_SYS);
        u32 ha[4], hb[4];
        unpack3(n, lo4, ha, hb);
        if (finfo & (1u << 19)) {
          const bool bd = ((lane < 32 ? mk[0] >> lane : mk[1] >> (lane - 32)) & 1) != 0;
#pragma unroll
          for (int j = 0; j < 4; ++j) {
            ha[j] = bd ? ova[j] : ha[j];
            hb[j] = bd ? ovb[j] : hb[j];
          }
        }
        *(u32x4*)(ua + lane * 16) = la;
        *(u32x4*)(ua + (nv + lane) * 16) = lb;
        *(u32x4*)(ua + 2176 + lane * 16) = u32x4{ha[0], ha[1], ha[2], ha[3]};
        *(u32x4*)(ua + 2176 + (nv + lane) * 16) = u32x4{hb[0], hb[1], hb[2], hb[3]};
      }
      bad = !ok2 || __builtin_amdgcn_readfirstlane(tm2) == call;
    }
    const u32 pc = __builtin_popcount(m);
    const u32 pos0 = wave_scan_incl(pc) - pc;
    const int si = wave >> 2, sj = wave & 3;
    const u32 obase = ((u32)(1024 * si + 32 * g + (lane >> 3)) * NN + 512 * sj + 64 * c + 8 * (lane & 7)) * 2;
    __amdgpu_buffer_rsrc_t rso = mk_rsrc(P.out, MM * NN * 2);
    asm volatile("" ::: "memory");
    __builtin_amdgcn_wave_barrier();
#pragma unroll
    for (int q = 0; q < 4; ++q) {
      const u32 mq = (m >> (8 * q)) & 0xff;
      u32 pos = pos0 + __builtin_popcount(q == 0 ? 0u : (m & ((1u << (8 * q)) - 1u)));
      pos = min(pos, 2048u);
      const u32x2 sel = *(const u32x2*)(smem + LDS_LUT + mq * 8);
      const u32* lw = (const u32*)(ua + (pos & ~3u));
      const u32* hw = (const u32*)(ua + 2176 + (pos & ~3u));
      const u32 l0 = lw[0], l1 = lw[1], l2 = lw[2];
      const u32 h0 = hw[0], h1 = hw[1], h2 = hw[2];
      const u32 la0 = __builtin_amdgcn_alignbyte(l1, l0, pos), la1 = __builtin_amdgcn_alignbyte(l2, l1, pos);
      const u32 ha0 = __builtin_amdgcn_alignbyte(h1, h0, pos), ha1 = __builtin_amdgcn_alignbyte(h2, h1, pos);
      const u32 eL0 = perm(la1, la0, sel[0]), eL1 = perm(la1, la0, sel[1]);
      const u32 eH0 = perm(ha1, ha0, sel[0]), eH1 = perm(ha1, ha0, sel[1]);
      u32x4 o = {perm(eH0, eL0, 0x05010400u), perm(eH0, eL0, 0x07030602u), perm(eH1, eL1, 0x05010400u), perm(eH1, eL1, 0x07030602u)};
      if (bad) o = u32x4{0x7fc07fc0u, 0x7fc07fc0u, 0x7fc07fc0u, 0x7fc07fc0u};
      __builtin_amdgcn_raw_buffer_store_b128(o, rso, obase + q * 8 * NN * 2, 0, OUT_AUX);
    }
  }
  if (tid == 0) __hip_atomic_store((gu32*)(myws + OFF_CNT + b * 128), call, __ATOMIC_RELAXED, __HIP_MEMORY_SCOPE_AGENT);
  STAMP(9);
}

}

extern "C" void kernel_launch(int rank, int nranks, ncclComm_t comm, hipStream_t stream,
                              void* const* d_in, const int* in_sizes, int n_in,
                              void* d_out, int out_size, const int* in_dtypes, int out_dtype,
                              void* const* d_out_all, void* const* d_ws_all, size_t ws_bytes) {
  if (nranks != NR || ws_bytes < WS_NEED || (unsigned)rank >= (unsigned)NR) {
    fprintf(stderr, "unexpected configuration\n");
    return;
  }
  static bool init[NR] = {};
  if (!init[rank]) {
    HIP_CHECK(hipFuncSetAttribute((const void*)gemm_ar_kernel, hipFuncAttributeMaxDynamicSharedMemorySize, LDS_BYTES));
    init[rank] = true;
  }
  Params p;
  p.x = (const uint16_t*)d_in[0];
  p.w = (const uint16_t*)d_in[1];
  for (int r = 0; r < NR; ++r) p.ws[r] = (char*)d_ws_all[r];
  p.myws = (char*)d_ws_all[rank];
  p.out = (char*)d_out;
  p.rank = rank;
  p.pad = 0;
  hipLaunchKernelGGL(gemm_ar_kernel, dim3(NWG), dim3(NT), LDS_BYTES, stream, p);
}
```

```cpp
#include <hip/hip_runtime.h>
#include <rccl/rccl.h>
#include <cstdio>
#include <cstdint>

#define HIP_CHECK(expr) do { hipError_t _e = (expr); if (_e != hipSuccess) { \
    fprintf(stderr, "HIP error %s at %s:%d\n", hipGetErrorString(_e), __FILE__, __LINE__); \
    return; } } while (0)

namespace {

typedef unsigned u32;
typedef unsigned long long u64;
typedef u32 u32x4 __attribute__((ext_vector_type(4)));
typedef u32 u32x2 __attribute__((ext_vector_type(2)));
typedef float f32x16 __attribute__((ext_vector_type(16)));
typedef float f32x4 __attribute__((ext_vector_type(4)));
typedef float f32x2 __attribute__((ext_vector_type(2)));
typedef __bf16 bf16x8 __attribute__((ext_vector_type(8)));
typedef __bf16 bf16x2 __attribute__((ext_vector_type(2)));
typedef unsigned short u16x2 __attribute__((ext_vector_type(2)));
typedef __attribute__((address_space(1))) u64 gu64;
typedef __attribute__((address_space(1))) u32 gu32;

constexpr int NR = 8;
constexpr int MM = 2048, NN = 2048, KL = 256;
constexpr int NWG = 256, NT = 512;

constexpr u32 OFF_CNT = 0;
constexpr u32 OFF_TMO = OFF_CNT + NWG * 128;
constexpr u32 OFF_WF = 65536;
constexpr u32 OFF_XF = OFF_WF + 64 * 128;
constexpr u32 OFF_OF = OFF_XF + 256 * 128;
constexpr u32 UNIT_B = 4096;
constexpr u32 OFF_WD = 1u << 20;
constexpr u32 OFF_XD = OFF_WD + 8 * 8 * 8 * UNIT_B;
constexpr u32 OT_B = 5248;
constexpr u32 OT_OVF = 3072, OT_BM = 5120;
constexpr u32 OFF_OD = OFF_XD + 8 * 32 * 4 * UNIT_B;
constexpr u32 WS_NEED = OFF_OD + 2048 * OT_B;
static_assert(OFF_OF + 2048 * 128 <= OFF_WD, "layout");

constexpr u64 DEADLINE = 50000000ull;
constexpr int AUX_SYS = 17;

constexpr u32 TW0 = 60, DW = 45;
constexpr u32 TX0 = 420, DX = 25;
constexpr u32 TO0 = 1210, DO = 26;
constexpr u32 LW = 120, LX = 120, LO = 120;
constexpr int GWFIRST = 12;
constexpr int OUT_AUX = 16;

constexpr int LDS_RED = 0;
constexpr int UAREA = 4352;
constexpr int LDS_STR = 65536;
constexpr int LDS_PK = LDS_STR + 4352;
constexpr int LDS_MSK = LDS_PK;
constexpr int LDS_OVF = LDS_PK + 3072;
constexpr int LDS_INFO = LDS_OVF + 2048;
constexpr int LDS_BAD = LDS_INFO + 32;
constexpr int LDS_PBM = LDS_BAD + 32;
constexpr int LDS_LUT = LDS_PBM + 16;
constexpr int LDS_BYTES = LDS_LUT + 2048;
static_assert(8 * UAREA <= 65536, "uarea");

struct Params {
  const uint16_t* x;
  const uint16_t* w;
  char* myws;
  char* out;
  int rank;
  int pad;
  char* ws[NR];
};

__device__ __forceinline__ __amdgpu_buffer_rsrc_t mk_rsrc(const void* p, u32 bytes) {
  return __builtin_amdgcn_make_buffer_rsrc((void*)p, 0, bytes, 0x00020000);
}
__device__ __forceinline__ u32 perm(u32 hi, u32 lo, u32 sel) { return __builtin_amdgcn_perm(hi, lo, sel); }
__device__ __forceinline__ u32 rotr4(u32 v) { return __builtin_amdgcn_alignbit(v, v, 4); }
__device__ __forceinline__ u32 rotl4(u32 v) { return __builtin_amdgcn_alignbit(v, v, 28); }
__device__ __forceinline__ u32 pkmax(u32 a, u32 b) {
  return __builtin_bit_cast(u32, __builtin_elementwise_max(__builtin_bit_cast(u16x2, a), __builtin_bit_cast(u16x2, b)));
}
template <int CTRL, int RM>
__device__ __forceinline__ u32 dpp0(u32 v) { return (u32)__builtin_amdgcn_update_dpp(0, (int)v, CTRL, RM, 0xf, false); }
__device__ __forceinline__ u32 wave_max(u32 v) {
  v = max(v, dpp0<0x111, 0xf>(v));
  v = max(v, dpp0<0x112, 0xf>(v));
  v = max(v, dpp0<0x114, 0xf>(v));
  v = max(v, dpp0<0x118, 0xf>(v));
  v = max(v, dpp0<0x142, 0xa>(v));
  v = max(v, dpp0<0x143, 0xc>(v));
  return (u32)__builtin_amdgcn_readlane((int)v, 63);
}
__device__ __forceinline__ u32 wave_scan_incl(u32 v) {
  v += dpp0<0x111, 0xf>(v);
  v += dpp0<0x112, 0xf>(v);
  v += dpp0<0x114, 0xf>(v);
  v += dpp0<0x118, 0xf>(v);
  v += dpp0<0x142, 0xa>(v);
  v += dpp0<0x143, 0xc>(v);
  return v;
}

__device__ __forceinline__ bool wait_flags(char* myws, u32 off, bool act, u32 call, int lane, u64& val, u64 tstart, u32 notbefore) {
  gu64* addr = (gu64*)(myws + off);
  while (__builtin_amdgcn_s_memrealtime() - tstart < (u64)notbefore) __builtin_amdgcn_s_sleep(2);
  const u64 t0 = __builtin_amdgcn_s_memrealtime();
  bool ok = true;
  for (;;) {
    u64 v = (u64)call;
    if (act) v = __hip_atomic_load(addr, __ATOMIC_RELAXED, __HIP_MEMORY_SCOPE_SYSTEM);
    val = v;
    const u64 pend = __builtin_amdgcn_ballot_w64(act && (u32)v < call);
    if (pend == 0) break;
    const u64 el = __builtin_amdgcn_s_memrealtime() - t0;
    if (el > 50000) {
      const u32 tm = (u32)__hip_atomic_load((gu64*)(myws + OFF_TMO), __ATOMIC_RELAXED, __HIP_MEMORY_SCOPE_AGENT);
      if (__builtin_amdgcn_readfirstlane(tm) == call || el > DEADLINE) {
        if (lane == 0) __hip_atomic_store((gu64*)(myws + OFF_TMO), (u64)call, __ATOMIC_RELAXED, __HIP_MEMORY_SCOPE_AGENT);
        ok = false;
        break;
      }
    }
    __builtin_amdgcn_s_sleep(4);
  }
  asm volatile("" ::: "memory");
  return ok;
}
__device__ __forceinline__ u32 read_tmo(char* myws) {
  return (u32)__hip_atomic_load((gu64*)(myws + OFF_TMO), __ATOMIC_RELAXED, __HIP_MEMORY_SCOPE_AGENT);
}

__device__ __forceinline__ u32 cvt_pk(float a, float b_) {
  f32x2 f = {a, b_};
  return __builtin_bit_cast(u32, __builtin_convertvector(f, bf16x2));
}

struct Packed { u32x4 la, lb, n, ovf; u64 bad; };

__device__ __forceinline__ void pack_hi(const u32 (&ha)[4], const u32 (&hb)[4], u32 lo, bool lane_on, Packed& pk) {
  const u32 lo4 = lo * 0x01010101u;
  u32 acc = 0x80808080u;
#pragma unroll
  for (int j = 0; j < 4; ++j) {
    const u32 da = (ha[j] | 0x80808080u) - lo4;
    const u32 db = (hb[j] | 0x80808080u) - lo4;
    acc &= da & db;
    const u32 xa = (ha[j] & 0x80808080u) | (da & 0x7f7f7f7fu);
    const u32 xb = (hb[j] & 0x80808080u) | (db & 0x7f7f7f7fu);
    pk.n[j] = xa | rotr4(xb);
  }
  const bool bad = lane_on && acc != 0x80808080u;
  pk.bad = __builtin_amdgcn_ballot_w64(bad);
  if (pk.bad != 0) {
#pragma unroll
    for (int j = 0; j < 4; ++j) {
      pk.n[j] = bad ? ha[j] : pk.n[j];
      pk.ovf[j] = hb[j];
    }
  }
}
__device__ __forceinline__ void unpack_hi(const u32x4& n, u32 lo4, u32 (&ha)[4], u32 (&hb)[4]) {
#pragma unroll
  for (int j = 0; j < 4; ++j) {
    ha[j] = (n[j] & 0x87878787u) + lo4;
    hb[j] = (rotl4(n[j]) & 0x87878787u) + lo4;
  }
}
typedef u32 u32x3 __attribute__((ext_vector_type(3)));
__device__ __forceinline__ u64 pack3(const u32 (&ha)[4], const u32 (&hb)[4], u32 lo, bool lane_on, u32 (&n)[3]) {
  const u32 lo4 = lo * 0x01010101u;
  u32 acc = 0x80808080u, sgn = 0, x[8];
#pragma unroll
  for (int j = 0; j < 4; ++j) {
    const u32 da = (ha[j] | 0x80808080u) - lo4;
    const u32 db = (hb[j] | 0x80808080u) - lo4;
    acc &= da & db;
    sgn |= ha[j] | hb[j];
    x[j] = da & 0x07070707u;
    x[4 + j] = db & 0x07070707u;
  }
  n[0] = x[0] | (x[1] << 3) | ((x[6] & 0x03030303u) << 6);
  n[1] = x[2] | (x[3] << 3) | ((x[7] & 0x03030303u) << 6);
  n[2] = x[4] | (x[5] << 3) | ((x[6] & 0x04040404u) << 4) | ((x[7] & 0x04040404u) << 5);
  const bool bad = lane_on && (acc != 0x80808080u || (sgn & 0x80808080u) != 0);
  return __builtin_amdgcn_ballot_w64(bad);
}
__device__ __forceinline__ void unpack3(const u32x3& n, u32 lo4, u32 (&ha)[4], u32 (&hb)[4]) {
  ha[0] = (n[0] & 0x07070707u) + lo4;
  ha[1] = ((n[0] >> 3) & 0x07070707u) + lo4;
  ha[2] = (n[1] & 0x07070707u) + lo4;
  ha[3] = ((n[1] >> 3) & 0x07070707u) + lo4;
  hb[0] = (n[2] & 0x07070707u) + lo4;
  hb[1] = ((n[2] >> 3) & 0x07070707u) + lo4;
  hb[2] = (((n[0] >> 6) & 0x03030303u) | ((n[2] >> 4) & 0x04040404u)) + lo4;
  hb[3] = (((n[1] >> 6) & 0x03030303u) | ((n[2] >> 5) & 0x04040404u)) + lo4;
}
__device__ __forceinline__ u32 lo_from_max(u32 m16) {
  const u32 e7 = m16 >> 8;
  return e7 > 7 ? e7 - 7 : 0;
}

__device__ __forceinline__ void timer_wait(u64 tstart, u32 dly) {
  while (__builtin_amdgcn_s_memrealtime() - tstart < (u64)dly) __builtin_amdgcn_s_sleep(1);
}

__device__ __forceinline__ u32 call_no(u32 cntv) {
  asm volatile("" : "+v"(cntv));
  return __builtin_amdgcn_readfirstlane(cntv) + 1;
}

template <int ND>
__device__ __forceinline__ void send_unit(const Params& P, const Packed& pk, u32 lo, int dst0, int dstep, u32 doff,
                                          u32 foff, u32 moff, u32 callv, int lane, u64 tstart, u32 dly) {
  timer_wait(tstart, dly);
#pragma unroll
  for (int d = 0; d < ND; ++d) {
    if (dst0 + d * dstep == P.rank) continue;
    __amdgpu_buffer_rsrc_t rs = mk_rsrc(P.ws[dst0 + d * dstep], WS_NEED);
    __builtin_amdgcn_raw_buffer_store_b128(pk.la, rs, doff + lane * 16, 0, AUX_SYS);
    __builtin_amdgcn_raw_buffer_store_b128(pk.lb, rs, doff + 1024 + lane * 16, 0, AUX_SYS);
    __builtin_amdgcn_raw_buffer_store_b128(pk.n, rs, doff + 2048 + lane * 16, 0, AUX_SYS);
  }
  if (pk.bad != 0) {
#pragma unroll
    for (int d = 0; d < ND; ++d) {
      if (dst0 + d * dstep == P.rank) continue;
      __amdgpu_buffer_rsrc_t rs = mk_rsrc(P.ws[dst0 + d * dstep], WS_NEED);
      if ((pk.bad >> lane) & 1) __builtin_amdgcn_raw_buffer_store_b128(pk.ovf, rs, doff + 3072 + lane * 16, 0, AUX_SYS);
      if (lane == 0) __builtin_amdgcn_raw_buffer_store_b64(u32x2{(u32)pk.bad, (u32)(pk.bad >> 32)}, rs, moff, 0, AUX_SYS);
    }
  }
  asm volatile("s_waitcnt vmcnt(0)" ::: "memory");
  const u32x2 f = {call_no(callv), lo | (pk.bad != 0 ? 0x80u : 0u)};
  if (lane == 0) {
#pragma unroll
    for (int d = 0; d < ND; ++d)
      if (dst0 + d * dstep != P.rank)
        __builtin_amdgcn_raw_buffer_store_b64(f, mk_rsrc(P.ws[dst0 + d * dstep], WS_NEED), foff, 0, AUX_SYS);
  }
}

__device__ __forceinline__ void send_x(const Params& P, const uint16_t* px, int g, int c, u32 call, int lane, u64 tstart, u32 dly) {
  const int rank = P.rank, h = lane >> 5, l31 = lane & 31;
  const int ih = c >> 2, u = c & 3;
  __amdgpu_buffer_rsrc_t rx = mk_rsrc(px, MM * KL * 2);
  const u32 voff = (u32)l31 * (KL * 2) + 256 * h;
  const u32 soff = (u32)(1024 * ih + 32 * g) * (KL * 2) + 64 * u;
  u32 r[16];
#pragma unroll
  for (int p = 0; p < 4; ++p) {
    const u32x4 t = __builtin_amdgcn_raw_buffer_load_b128(rx, voff + p * 16, soff, 0);
#pragma unroll
    for (int q = 0; q < 4; ++q) r[4 * p + q] = t[q];
  }
  u32 m = 0;
#pragma unroll
  for (int t = 0; t < 16; ++t) m = pkmax(m, r[t] & 0x7fff7fffu);
  const u32 lo = lo_from_max(wave_max(max(m & 0xffffu, m >> 16)));
  Packed pk;
  u32 ha[4], hb[4];
#pragma unroll
  for (int j = 0; j < 4; ++j) {
    pk.la[j] = perm(r[2 * j + 1], r[2 * j], 0x06040200u);
    ha[j] = perm(r[2 * j + 1], r[2 * j], 0x07050301u);
    pk.lb[j] = perm(r[8 + 2 * j + 1], r[8 + 2 * j], 0x06040200u);
    hb[j] = perm(r[8 + 2 * j + 1], r[8 + 2 * j], 0x07050301u);
  }
  pack_hi(ha, hb, lo, true, pk);
  const u32 fl = OFF_XF + (rank * 32 + g) * 128;
  send_unit<4>(P, pk, lo, 4 * ih, 1, OFF_XD + (u32)((rank * 32 + g) * 4 + u) * UNIT_B, fl + u * 8, fl + 32 + u * 8, call, lane, tstart, dly);
}

__device__ __forceinline__ void send_w(const Params& P, const uint16_t* pw, int jd, int cd, int u8, u32 call, int lane, u64 tstart, u32 dly) {
  const int rank = P.rank, h = lane >> 5, l31 = lane & 31;
  const int cgl = 8 * jd + cd;
  __amdgpu_buffer_rsrc_t rw = mk_rsrc(pw, KL * NN * 2);
  const u32 voff = (u32)(128 * h) * (NN * 2) + 4 * l31;
  const u32 soff = (u32)(16 * u8) * (NN * 2) + 128 * cgl;
  u32 r[16];
#pragma unroll
  for (int k = 0; k < 16; ++k) r[k] = __builtin_amdgcn_raw_buffer_load_b32(rw, voff, soff + k * (NN * 2), 0);
  u32 m = 0;
#pragma unroll
  for (int t = 0; t < 16; ++t) m = pkmax(m, r[t] & 0x7fff7fffu);
  const u32 lo = lo_from_max(wave_max(max(m & 0xffffu, m >> 16)));
  Packed pk;
  u32 ha[4], hb[4];
#pragma unroll
  for (int j = 0; j < 4; ++j) {
    const u32 t0 = perm(r[4 * j + 1], r[4 * j], 0x05010400u), t1 = perm(r[4 * j + 1], r[4 * j], 0x07030602u);
    const u32 t2 = perm(r[4 * j + 3], r[4 * j + 2], 0x05010400u), t3 = perm(r[4 * j + 3], r[4 * j + 2], 0x07030602u);
    pk.la[j] = perm(t2, t0, 0x05040100u);
    ha[j] = perm(t2, t0, 0x07060302u);
    pk.lb[j] = perm(t3, t1, 0x05040100u);
    hb[j] = perm(t3, t1, 0x07060302u);
  }
  pack_hi(ha, hb, lo, true, pk);
  const u32 fl = OFF_WF + (rank * 8 + cd) * 128 + (u8 >> 2) * 64 + (u8 & 3) * 8;
  send_unit<2>(P, pk, lo, jd, 4, OFF_WD + (u32)((rank * 8 + cd) * 8 + u8) * UNIT_B, fl, fl + 32, call, lane, tstart, dly);
}

__device__ __forceinline__ void decode_unit(const u32x4& la, const u32x4& lb, const u32x4& n, const u32x4& ov, u32 info,
                                            u64 badmask, u32 (&o)[16]) {
  const u32 lo4 = (info & 0x7f) * 0x01010101u;
  u32 ha[4], hb[4];
  unpack_hi(n, lo4, ha, hb);
  if (info & 0x80) {
    const bool bad = __builtin_amdgcn_inverse_ballot_w64(badmask);
#pragma unroll
    for (int j = 0; j < 4; ++j) {
      ha[j] = bad ? n[j] : ha[j];
      hb[j] = bad ? ov[j] : hb[j];
    }
  }
#pragma unroll
  for (int j = 0; j < 4; ++j) {
    o[2 * j] = perm(ha[j], la[j], 0x05010400u);
    o[2 * j + 1] = perm(ha[j], la[j], 0x07030602u);
    o[8 + 2 * j] = perm(hb[j], lb[j], 0x05010400u);
    o[8 + 2 * j + 1] = perm(hb[j], lb[j], 0x07030602u);
  }
}
__device__ __forceinline__ u64 rdlane64(const u32x2& v, int l) {
  return (u64)(u32)__builtin_amdgcn_readlane(v[0], l) | ((u64)(u32)__builtin_amdgcn_readlane(v[1], l) << 32);
}

__global__ __launch_bounds__(NT, 2) void gemm_ar_kernel(Params P) {
  extern __shared__ __attribute__((aligned(16))) char smem[];
  const int tid = threadIdx.x;
  const int lane = tid & 63;
  const int wave = __builtin_amdgcn_readfirstlane(tid >> 6);
  const int b = blockIdx.x;
  int rank = P.rank;
  const int g = b >> 3, c = b & 7;
  char* myws = P.myws;
  const int h = lane >> 5, l31 = lane & 31;
  const uint16_t* pw = P.w;
  const uint16_t* px = P.x;
  asm volatile("" : "+s"(pw), "+s"(px), "+s"(myws), "+s"(rank));
  const u64 tstart = __builtin_amdgcn_s_memrealtime();

  const u32 callv = __hip_atomic_load((gu32*)(myws + OFF_CNT + b * 128), __ATOMIC_RELAXED, __HIP_MEMORY_SCOPE_AGENT);
  bool ok = true;

  __amdgpu_buffer_rsrc_t rsme = mk_rsrc(myws, WS_NEED);
  f32x16 acc0 = {}, acc1 = {};
  u32 tmv = 0;
  if (wave == rank) {
    send_w(P, pw, g & 3, g >> 2, c, callv, lane, tstart, TW0 + (g >> 2) * DW);
    if (g < GWFIRST) send_x(P, px, g, c, callv, lane, tstart, TX0 + g * DX);
    {
      __amdgpu_buffer_rsrc_t rw = mk_rsrc(pw, KL * NN * 2);
      __amdgpu_buffer_rsrc_t rx = mk_rsrc(px, MM * KL * 2);
      const u32 wv = (u32)(128 * h) * (NN * 2) + 4 * l31;
      const u32 ws0 = (512 * (rank & 3) + 64 * c) * 2;
      const u32 xv = (u32)l31 * (KL * 2) + 256 * h;
      const u32 xs0 = (u32)(1024 * (rank >> 2) + 32 * g) * (KL * 2);
      u32 r[4][32];
      u32x4 xf[4][4];
      auto ld = [&](int q) {
#pragma unroll
        for (int k = 0; k < 32; ++k) r[q][k] = __builtin_amdgcn_raw_buffer_load_b32(rw, wv, ws0 + (32 * q + k) * (NN * 2), 0);
#pragma unroll
        for (int m = 0; m < 4; ++m) xf[q][m] = __builtin_amdgcn_raw_buffer_load_b128(rx, xv + (4 * q + m) * 16, xs0, 0);
      };
      auto mm = [&](int q) {
#pragma unroll
        for (int e = 0; e < 4; ++e) {
          u32x4 w0, w1;
#pragma unroll
          for (int t = 0; t < 4; ++t) {
            w0[t] = perm(r[q][8 * e + 2 * t + 1], r[q][8 * e + 2 * t], 0x05040100u);
            w1[t] = perm(r[q][8 * e + 2 * t + 1], r[q][8 * e + 2 * t], 0x07060302u);
          }
          acc0 = __builtin_amdgcn_mfma_f32_32x32x16_bf16(__builtin_bit_cast(bf16x8, xf[q][e]), __builtin_bit_cast(bf16x8, w0), acc0, 0, 0, 0);
          acc1 = __builtin_amdgcn_mfma_f32_32x32x16_bf16(__builtin_bit_cast(bf16x8, xf[q][e]), __builtin_bit_cast(bf16x8, w1), acc1, 0, 0, 0);
        }
      };
      ld(0);
      __builtin_amdgcn_sched_barrier(0);
      {
#pragma unroll
        for (int k = 0; k < 24; ++k) r[1][k] = __builtin_amdgcn_raw_buffer_load_b32(rw, wv, ws0 + (32 + k) * (NN * 2), 0);
      }
      __builtin_amdgcn_sched_barrier(0);
      mm(0);
      __builtin_amdgcn_sched_barrier(0);
      {
#pragma unroll
        for (int k = 24; k < 32; ++k) r[1][k] = __builtin_amdgcn_raw_buffer_load_b32(rw, wv, ws0 + (32 + k) * (NN * 2), 0);
#pragma unroll
        for (int m = 0; m < 4; ++m) xf[1][m] = __builtin_amdgcn_raw_buffer_load_b128(rx, xv + (4 + m) * 16, xs0, 0);
#pragma unroll
        for (int k = 0; k < 24; ++k) r[2][k] = __builtin_amdgcn_raw_buffer_load_b32(rw, wv, ws0 + (64 + k) * (NN * 2), 0);
      }
      __builtin_amdgcn_sched_barrier(0);
      mm(1);
      __builtin_amdgcn_sched_barrier(0);
      {
#pragma unroll
        for (int k = 24; k < 32; ++k) r[2][k] = __builtin_amdgcn_raw_buffer_load_b32(rw, wv, ws0 + (64 + k) * (NN * 2), 0);
#pragma unroll
        for (int m = 0; m < 4; ++m) xf[2][m] = __builtin_amdgcn_raw_buffer_load_b128(rx, xv + (8 + m) * 16, xs0, 0);
#pragma unroll
        for (int k = 0; k < 24; ++k) r[3][k] = __builtin_amdgcn_raw_buffer_load_b32(rw, wv, ws0 + (96 + k) * (NN * 2), 0);
      }
      __builtin_amdgcn_sched_barrier(0);
      mm(2);
      __builtin_amdgcn_sched_barrier(0);
      {
#pragma unroll
        for (int k = 24; k < 32; ++k) r[3][k] = __builtin_amdgcn_raw_buffer_load_b32(rw, wv, ws0 + (96 + k) * (NN * 2), 0);
#pragma unroll
        for (int m = 0; m < 4; ++m) xf[3][m] = __builtin_amdgcn_raw_buffer_load_b128(rx, xv + (12 + m) * 16, xs0, 0);
      }
      __builtin_amdgcn_sched_barrier(0);
      mm(3);
    }
    if (g >= GWFIRST) send_x(P, px, g, c, callv, lane, tstart, TX0 + g * DX);
    tmv = read_tmo(myws);
  } else {
    const u32 lw4 = (wave - rank - 1) & 7;
    if (lw4 < 4) {
      const u32 m = lw4 * 64 + lane;
      u32 sel[2] = {0u, 0u};
      u32 cnt = 0;
#pragma unroll
      for (int j = 0; j < 8; ++j) {
        const u32 bit = (m >> j) & 1;
        const u32 s = bit ? cnt : 0x0cu;
        sel[j >> 2] |= s << (8 * (j & 3));
        cnt += bit;
      }
      *(u32x2*)(smem + LDS_LUT + m * 8) = u32x2{sel[0], sel[1]};
    }
    const u32 call = call_no(callv);
    u32x4 wf[2][16];
  {
    u64 fv;
    const u32 fl = OFF_WF + (wave * 8 + c) * 128;
    ok = wait_flags(myws, fl + (lane >> 2) * 64 + (lane & 3) * 8, lane < 8, call, lane, fv, tstart, max(TW0 + c * DW, 100u) + LW) && ok;
    const u32 off = OFF_WD + (u32)((wave * 8 + c) * 8) * UNIT_B;
    const u32 finfo = (u32)(fv >> 32);
    const u32 ovm = (u32)__builtin_amdgcn_ballot_w64(lane < 8 && (finfo & 0x80) != 0);
    u32x4 pa[8], pb[8], pn[8], po[8];
    u32x2 mk = {0u, 0u};
    if (ovm != 0) {
      mk = __builtin_amdgcn_raw_buffer_load_b64(rsme, fl + ((lane >> 2) & 1) * 64 + 32 + (lane & 3) * 8, 0, AUX_SYS);
#pragma unroll
      for (int u8 = 0; u8 < 8; ++u8) {
        po[u8] = u32x4{0u, 0u, 0u, 0u};
        if ((ovm >> u8) & 1) po[u8] = __builtin_amdgcn_raw_buffer_load_b128(rsme, off + u8 * UNIT_B + 3072 + lane * 16, 0, AUX_SYS);
      }
    } else {
#pragma unroll
      for (int u8 = 0; u8 < 8; ++u8) po[u8] = u32x4{0u, 0u, 0u, 0u};
    }
#pragma unroll
    for (int u8 = 0; u8 < 8; ++u8) {
      pa[u8] = __builtin_amdgcn_raw_buffer_load_b128(rsme, off + u8 * UNIT_B + lane * 16, 0, AUX_SYS);
      pb[u8] = __builtin_amdgcn_raw_buffer_load_b128(rsme, off + u8 * UNIT_B + 1024 + lane * 16, 0, AUX_SYS);
      pn[u8] = __builtin_amdgcn_raw_buffer_load_b128(rsme, off + u8 * UNIT_B + 2048 + lane * 16, 0, AUX_SYS);
    }
    __builtin_amdgcn_sched_barrier(0);
#pragma unroll
    for (int u8 = 0; u8 < 8; ++u8) {
      const u32 info = __builtin_amdgcn_readlane(finfo, u8);
      u32 o[16];
      decode_unit(pa[u8], pb[u8], pn[u8], po[u8], info, rdlane64(mk, u8), o);
#pragma unroll
      for (int e = 0; e < 2; ++e)
#pragma unroll
        for (int q = 0; q < 4; ++q) {
          wf[0][2 * u8 + e][q] = o[4 * e + q];
          wf[1][2 * u8 + e][q] = o[8 + 4 * e + q];
        }
    }
  }
  {
    u64 fv;
    const u32 fl = OFF_XF + (wave * 32 + g) * 128;
    ok = wait_flags(myws, fl + lane * 8, lane < 4, call, lane, fv, tstart, TX0 + g * DX + LX) && ok;
    tmv = read_tmo(myws);
    const u32 off = OFF_XD + (u32)((wave * 32 + g) * 4) * UNIT_B;
    const u32 finfo = (u32)(fv >> 32);
    const u32 ovm = (u32)__builtin_amdgcn_ballot_w64(lane < 4 && (finfo & 0x80) != 0);
    u32x4 pa[4], pb[4], pn[4], po[4];
    u32x2 mk = {0u, 0u};
    if (ovm != 0) {
      mk = __builtin_amdgcn_raw_buffer_load_b64(rsme, fl + 32 + (lane & 3) * 8, 0, AUX_SYS);
#pragma unroll
      for (int u = 0; u < 4; ++u) {
        po[u] = u32x4{0u, 0u, 0u, 0u};
        if ((ovm >> u) & 1) po[u] = __builtin_amdgcn_raw_buffer_load_b128(rsme, off + u * UNIT_B + 3072 + lane * 16, 0, AUX_SYS);
      }
    } else {
#pragma unroll
      for (int u = 0; u < 4; ++u) po[u] = u32x4{0u, 0u, 0u, 0u};
    }
#pragma unroll
    for (int u = 0; u < 4; ++u) {
      pa[u] = __builtin_amdgcn_raw_buffer_load_b128(rsme, off + u * UNIT_B + lane * 16, 0, AUX_SYS);
      pb[u] = __builtin_amdgcn_raw_buffer_load_b128(rsme, off + u * UNIT_B + 1024 + lane * 16, 0, AUX_SYS);
      pn[u] = __builtin_amdgcn_raw_buffer_load_b128(rsme, off + u * UNIT_B + 2048 + lane * 16, 0, AUX_SYS);
    }
    __builtin_amdgcn_sched_barrier(0);
#pragma unroll
    for (int u = 0; u < 4; ++u) {
      const u32 info = __builtin_amdgcn_readlane(finfo, u);
      u32 o[16];
      decode_unit(pa[u], pb[u], pn[u], po[u], info, rdlane64(mk, u), o);
#pragma unroll
      for (int e = 0; e < 4; ++e) {
        const u32x4 xf = {o[4 * e], o[4 * e + 1], o[4 * e + 2], o[4 * e + 3]};
        const int m = 4 * u + e;
        acc0 = __builtin_amdgcn_mfma_f32_32x32x16_bf16(__builtin_bit_cast(bf16x8, xf), __builtin_bit_cast(bf16x8, wf[0][m]), acc0, 0, 0, 0);
        acc1 = __builtin_amdgcn_mfma_f32_32x32x16_bf16(__builtin_bit_cast(bf16x8, xf), __builtin_bit_cast(bf16x8, wf[1][m]), acc1, 0, 0, 0);
      }
    }
  }
  }
  const u32 call = call_no(callv);
  {
    char* red = smem + LDS_RED + wave * 8192;
#pragma unroll
    for (int r = 0; r < 16; ++r) {
      const int row = (r & 3) + 8 * (r >> 2) + 4 * h;
      const int off = (row * 256 + l31 * 8) ^ (((row >> 3) & 1) << 7);
      *(f32x2*)(red + off) = f32x2{acc0[r], acc1[r]};
    }
    const bool bad = !ok || __builtin_amdgcn_readfirstlane(tmv) == call;
    if (lane == 0) ((u32*)(smem + LDS_BAD))[wave] = bad ? 1u : 0u;
  }
  __syncthreads();
  u32 o0, o1, nib, lpre;
  {
    const int cg = lane >> 3, q = (lane >> 1) & 3, hf = lane & 1;
    const int row = 8 * q + wave;
    const int off = (row * 256 + cg * 32 + hf * 16) ^ ((q & 1) << 7);
    f32x4 s = *(const f32x4*)(smem + LDS_RED + off);
#pragma unroll
    for (int w2 = 1; w2 < 8; ++w2) s += *(const f32x4*)(smem + LDS_RED + w2 * 8192 + off);
    const u32x4 bw0 = *(const u32x4*)(smem + LDS_BAD), bw1 = *(const u32x4*)(smem + LDS_BAD + 16);
    const bool bad = (bw0[0] | bw0[1] | bw0[2] | bw0[3] | bw1[0] | bw1[1] | bw1[2] | bw1[3]) != 0;
    float v[4];
#pragma unroll
    for (int e = 0; e < 4; ++e) v[e] = !(s[e] <= 0.f) ? s[e] : 0.f;
    o0 = cvt_pk(v[0], v[1]);
    o1 = cvt_pk(v[2], v[3]);
    if (bad) { o0 = 0x7fc07fc0u; o1 = 0x7fc07fc0u; }
    nib = ((o0 & 0xffffu) ? 1u : 0u) | ((o0 >> 16) ? 2u : 0u) | ((o1 & 0xffffu) ? 4u : 0u) | ((o1 >> 16) ? 8u : 0u);
    const u32 cnt = __builtin_popcount(nib);
    const u32 incl = wave_scan_incl(cnt);
    lpre = incl - cnt;
    const u32 tot = __builtin_amdgcn_readlane(incl, 63);
    const u32 pm = pkmax(o0 & 0x7fff7fffu, o1 & 0x7fff7fffu);
    const u32 mx = wave_max(max(pm & 0xffffu, pm >> 16));
    if (lane == 0) ((u32*)(smem + LDS_INFO))[wave] = tot | (mx << 16);
  }
  __syncthreads();
  u32 count, mx16;
  {
    const u32x4 i0 = *(const u32x4*)(smem + LDS_INFO), i1 = *(const u32x4*)(smem + LDS_INFO + 16);
    u32 pre = 0, tot = 0, mx = 0;
#pragma unroll
    for (int w2 = 0; w2 < 8; ++w2) {
      const u32 iv = w2 < 4 ? i0[w2 & 3] : i1[w2 & 3];
      if (w2 == wave) pre = tot;
      tot += iv & 0xffffu;
      mx = max(mx, iv >> 16);
    }
    count = __builtin_amdgcn_readfirstlane(tot);
    mx16 = __builtin_amdgcn_readfirstlane(mx);
    uint16_t* str = (uint16_t*)(smem + LDS_STR);
    u32 p = pre + lpre;
    if (nib & 1) str[p] = (uint16_t)o0;
    p += nib & 1;
    if (nib & 2) str[p] = (uint16_t)(o0 >> 16);
    p += (nib >> 1) & 1;
    if (nib & 4) str[p] = (uint16_t)o1;
    p += (nib >> 2) & 1;
    if (nib & 8) str[p] = (uint16_t)(o1 >> 16);
    const u32 nnext = (u32)__builtin_amdgcn_update_dpp(0, (int)nib, 0x101  , 0xf, 0xf, false);
    if ((lane & 1) == 0) *(uint8_t*)(smem + LDS_MSK + wave * 32 + (lane >> 1)) = (uint8_t)(nib | (nnext << 4));
    if (wave == 0) str[count + lane] = (uint16_t)mx16;
  }
  __syncthreads();
  const u32 nvs = (count + 31) >> 5;
  const u32 los = lo_from_max(mx16);
  if (wave == rank) {
    const u32 nv = nvs;
    const bool on = (u32)lane < nv;
    const char* sa = smem + LDS_STR + lane * 32;
    const char* sb = on ? sa + nv * 32 : sa;
    u32 r[16];
    {
      const u32x4 t0 = *(const u32x4*)(sa), t1 = *(const u32x4*)(sa + 16);
      const u32x4 t2 = *(const u32x4*)(sb), t3 = *(const u32x4*)(sb + 16);
#pragma unroll
      for (int q = 0; q < 4; ++q) { r[q] = t0[q]; r[4 + q] = t1[q]; r[8 + q] = t2[q]; r[12 + q] = t3[q]; }
    }
    u32x4 la, lb;
    u32 ha[4], hb[4], n[3];
#pragma unroll
    for (int j = 0; j < 4; ++j) {
      la[j] = perm(r[2 * j + 1], r[2 * j], 0x06040200u);
      ha[j] = perm(r[2 * j + 1], r[2 * j], 0x07050301u);
      lb[j] = perm(r[8 + 2 * j + 1], r[8 + 2 * j], 0x06040200u);
      hb[j] = perm(r[8 + 2 * j + 1], r[8 + 2 * j], 0x07050301u);
    }
    const u64 bm = pack3(ha, hb, los, on, n);
    char* ua = smem + LDS_RED + wave * UAREA;
    char* pk = smem + LDS_PK + 256;
    if (on) {
      *(u32x4*)(pk + lane * 16) = la;
      *(u32x4*)(pk + (nv + lane) * 16) = lb;
      u32* pn = (u32*)(pk + nv * 32 + lane * 12);
      pn[0] = n[0]; pn[1] = n[1]; pn[2] = n[2];
      *(u32x4*)(ua + lane * 16) = la;
      *(u32x4*)(ua + (nv + lane) * 16) = lb;
      *(u32x4*)(ua + 2176 + lane * 16) = u32x4{ha[0], ha[1], ha[2], ha[3]};
      *(u32x4*)(ua + 2176 + (nv + lane) * 16) = u32x4{hb[0], hb[1], hb[2], hb[3]};
    }
    if (bm != 0) {
      *(u32x4*)(smem + LDS_OVF + lane * 32) = u32x4{ha[0], ha[1], ha[2], ha[3]};
      *(u32x4*)(smem + LDS_OVF + lane * 32 + 16) = u32x4{hb[0], hb[1], hb[2], hb[3]};
    }
    if (lane == 0) *(u32x2*)(smem + LDS_PBM) = u32x2{(u32)bm, (u32)(bm >> 32)};
  }
  __syncthreads();
  if (wave != rank) {
    const u32x2 bmv = *(const u32x2*)(smem + LDS_PBM);
    const u64 bm = ((u64)(u32)__builtin_amdgcn_readfirstlane(bmv[1]) << 32) | (u32)__builtin_amdgcn_readfirstlane(bmv[0]);
    const u32 tb = 256 + 44 * nvs;
    __amdgpu_buffer_rsrc_t rs = mk_rsrc(P.ws[wave], WS_NEED);
    const u32 slot = OFF_OD + (u32)(rank * 256 + g * 8 + c) * OT_B;
    u32x4 v[3];
#pragma unroll
    for (int k = 0; k < 3; ++k) v[k] = *(const u32x4*)(smem + LDS_PK + (k * 64 + lane) * 16);
    timer_wait(tstart, TO0 + g * DO);
#pragma unroll
    for (int k = 0; k < 3; ++k)
      if ((u32)(k * 64 + lane) * 16 < tb) __builtin_amdgcn_raw_buffer_store_b128(v[k], rs, slot + (k * 64 + lane) * 16, 0, AUX_SYS);
    if (bm != 0) {
      if ((bm >> lane) & 1) {
        __builtin_amdgcn_raw_buffer_store_b128(*(const u32x4*)(smem + LDS_OVF + lane * 32), rs, slot + OT_OVF + lane * 32, 0, AUX_SYS);
        __builtin_amdgcn_raw_buffer_store_b128(*(const u32x4*)(smem + LDS_OVF + lane * 32 + 16), rs, slot + OT_OVF + lane * 32 + 16, 0, AUX_SYS);
      }
      if (lane == 0) __builtin_amdgcn_raw_buffer_store_b64(bmv, rs, slot + OT_BM, 0, AUX_SYS);
    }
    asm volatile("s_waitcnt vmcnt(0)" ::: "memory");
    if (lane == 0) {
      const u32x2 f = {call, count | (los << 12) | (bm != 0 ? 1u << 19 : 0u)};
      __builtin_amdgcn_raw_buffer_store_b64(f, rs, OFF_OF + (rank * 256 + g * 8 + c) * 128, 0, AUX_SYS);
    }
  }
  {
    char* ua = smem + LDS_RED + wave * UAREA;
    u32 m;
    bool bad = false;
    if (wave == rank) {
      m = *(const u32*)(smem + LDS_MSK + lane * 4);
    } else {
      u64 fv;
      const bool ok2 = wait_flags(myws, OFF_OF + (wave * 256 + g * 8 + c) * 128, lane == 0, call, lane, fv, tstart, TO0 + g * DO + LO);
      const u32 tm2 = read_tmo(myws);
      const u32 finfo = __builtin_amdgcn_readlane((u32)(fv >> 32), 0);
      const u32 cnt2 = ok2 ? (finfo & 0xfffu) : 0u;
      const u32 lo4 = ((finfo >> 12) & 0x7f) * 0x01010101u;
      const u32 nv = (cnt2 + 31) >> 5;
      const u32 slot = OFF_OD + (u32)(wave * 256 + g * 8 + c) * OT_B;
      m = __builtin_amdgcn_raw_buffer_load_b32(rsme, slot + lane * 4, 0, AUX_SYS);
      u32x4 ova = {0u, 0u, 0u, 0u}, ovb = {0u, 0u, 0u, 0u};
      u32x2 mk = {0u, 0u};
      if (finfo & (1u << 19)) {
        ova = __builtin_amdgcn_raw_buffer_load_b128(rsme, slot + OT_OVF + lane * 32, 0, AUX_SYS);
        ovb = __builtin_amdgcn_raw_buffer_load_b128(rsme, slot + OT_OVF + lane * 32 + 16, 0, AUX_SYS);
        mk = __builtin_amdgcn_raw_buffer_load_b64(rsme, slot + OT_BM, 0, AUX_SYS);
      }
      if ((u32)lane < nv) {
        const u32x4 la = __builtin_amdgcn_raw_buffer_load_b128(rsme, slot + 256 + lane * 16, 0, AUX_SYS);
        const u32x4 lb = __builtin_amdgcn_raw_buffer_load_b128(rsme, slot + 256 + (nv + lane) * 16, 0, AUX_SYS);
        const u32x3 n = __builtin_amdgcn_raw_buffer_load_b96(rsme, slot + 256 + nv * 32 + lane * 12, 0, AUX_SYS);
        u32 ha[4], hb[4];
        unpack3(n, lo4, ha, hb);
        if (finfo & (1u << 19)) {
          const bool bd = ((lane < 32 ? mk[0] >> lane : mk[1] >> (lane - 32)) & 1) != 0;
#pragma unroll
          for (int j = 0; j < 4; ++j) {
            ha[j] = bd ? ova[j] : ha[j];
            hb[j] = bd ? ovb[j] : hb[j];
          }
        }
        *(u32x4*)(ua + lane * 16) = la;
        *(u32x4*)(ua + (nv + lane) * 16) = lb;
        *(u32x4*)(ua + 2176 + lane * 16) = u32x4{ha[0], ha[1], ha[2], ha[3]};
        *(u32x4*)(ua + 2176 + (nv + lane) * 16) = u32x4{hb[0], hb[1], hb[2], hb[3]};
      }
      bad = !ok2 || __builtin_amdgcn_readfirstlane(tm2) == call;
    }
    const u32 pc = __builtin_popcount(m);
    const u32 pos0 = wave_scan_incl(pc) - pc;
    const int si = wave >> 2, sj = wave & 3;
    const u32 obase = ((u32)(1024 * si + 32 * g + (lane >> 3)) * NN + 512 * sj + 64 * c + 8 * (lane & 7)) * 2;
    __amdgpu_buffer_rsrc_t rso = mk_rsrc(P.out, MM * NN * 2);
    asm volatile("" ::: "memory");
    __builtin_amdgcn_wave_barrier();
#pragma unroll
    for (int q = 0; q < 4; ++q) {
      const u32 mq = (m >> (8 * q)) & 0xff;
      u32 pos = pos0 + __builtin_popcount(q == 0 ? 0u : (m & ((1u << (8 * q)) - 1u)));
      pos = min(pos, 2048u);
      const u32x2 sel = *(const u32x2*)(smem + LDS_LUT + mq * 8);
      const u32* lwin = (const u32*)(ua + (pos & ~3u));
      const u32* hwin = (const u32*)(ua + 2176 + (pos & ~3u));
      const u32 l0 = lwin[0], l1 = lwin[1], l2 = lwin[2];
      const u32 h0 = hwin[0], h1 = hwin[1], h2 = hwin[2];
      const u32 la0 = __builtin_amdgcn_alignbyte(l1, l0, pos), la1 = __builtin_amdgcn_alignbyte(l2, l1, pos);
      const u32 ha0 = __builtin_amdgcn_alignbyte(h1, h0, pos), ha1 = __builtin_amdgcn_alignbyte(h2, h1, pos);
      const u32 eL0 = perm(la1, la0, sel[0]), eL1 = perm(la1, la0, sel[1]);
      const u32 eH0 = perm(ha1, ha0, sel[0]), eH1 = perm(ha1, ha0, sel[1]);
      u32x4 o = {perm(eH0, eL0, 0x05010400u), perm(eH0, eL0, 0x07030602u), perm(eH1, eL1, 0x05010400u), perm(eH1, eL1, 0x07030602u)};
      if (bad) o = u32x4{0x7fc07fc0u, 0x7fc07fc0u, 0x7fc07fc0u, 0x7fc07fc0u};
      __builtin_amdgcn_raw_buffer_store_b128(o, rso, obase + q * 8 * NN * 2, 0, OUT_AUX);
    }
  }
  if (tid == 0) __hip_atomic_store((gu32*)(myws + OFF_CNT + b * 128), call, __ATOMIC_RELAXED, __HIP_MEMORY_SCOPE_AGENT);
}

}

extern "C" void kernel_launch(int rank, int nranks, ncclComm_t comm, hipStream_t stream,
                              void* const* d_in, const int* in_sizes, int n_in,
                              void* d_out, int out_size, const int* in_dtypes, int out_dtype,
                              void* const* d_out_all, void* const* d_ws_all, size_t ws_bytes) {
  if (nranks != NR || ws_bytes < WS_NEED || (unsigned)rank >= (unsigned)NR) {
    fprintf(stderr, "unexpected configuration\n");
    return;
  }
  static bool init[NR] = {};
  if (!init[rank]) {
    HIP_CHECK(hipFuncSetAttribute((const void*)gemm_ar_kernel, hipFuncAttributeMaxDynamicSharedMemorySize, LDS_BYTES));
    init[rank] = true;
  }
  Params p;
  p.x = (const uint16_t*)d_in[0];
  p.w = (const uint16_t*)d_in[1];
  for (int r = 0; r < NR; ++r) p.ws[r] = (char*)d_ws_all[r];
  p.myws = (char*)d_ws_all[rank];
  p.out = (char*)d_out;
  p.rank = rank;
  p.pad = 0;
  hipLaunchKernelGGL(gemm_ar_kernel, dim3(NWG), dim3(NT), LDS_BYTES, stream, p);
}
```
